```python
import numpy as np

kernel_name = "gpu_ag_gemm_m4096_k4096_n16384_e4m3e4m3_dqrelu"


ABSMAX_RTOL = 0.03

COMPUTE_DTYPE = "float32"
EPILOGUE = "dequant_relu"
IN_QUANT = None
OUT_QUANT = None
IN_FP8 = "e4m3e4m3"
IN_DTYPES = {"x": "e4m3", "w_mat": "e4m3",
             "scale_x": "float32", "scale_w": "float32"}
WORLD_SIZE = 8

def _snap_fp8(x, mant_bits, max_abs, min_normal):
    a = np.abs(x.astype(np.float64))
    _m, _e = np.frexp(a)
    q = np.where(a >= min_normal, np.ldexp(1.0, _e - 1 - mant_bits),
                 min_normal * 2.0 ** -mant_bits)
    snapped = np.minimum(np.round(a / q) * q, max_abs)
    return (np.sign(x) * snapped).astype(np.float32)

def setup_inputs():

    x = _snap_fp8(np.random.randn(4096, 4096).astype(np.float32), 3, 448.0, 2.0 ** -6)
    w_mat = _snap_fp8(np.random.randn(4096, 16384).astype(np.float32), 3, 448.0, 2.0 ** -6)
    scale_x = (0.02 * (1.0 + np.random.rand(1))).astype(np.float32)
    scale_w = (0.02 * (1.0 + np.random.rand(1))).astype(np.float32)
    return {"x": x, "w_mat": w_mat, "scale_x": scale_x, "scale_w": scale_w}

def reference(x, w_mat, scale_x, scale_w):
    acc = (x @ w_mat).astype(np.float32)
    y = (acc * (scale_x * scale_w)).astype(np.float32)
    return np.maximum(y, 0.0).astype(np.float32)
```

```cpp
#include <hip/hip_runtime.h>
#include <rccl/rccl.h>
#include <cstdio>
#include <utility>

#define HIP_CHECK(expr) do { hipError_t _e = (expr); if (_e != hipSuccess) { \
    fprintf(stderr, "HIP error %s at %s:%d\n", hipGetErrorString(_e), __FILE__, __LINE__); \
    return; } } while (0)

#define AG_OUT_AUX 0
#define AG_D 4
#define AG_NS 80
#define AG_NA 48
#define AG_TH 24
#define AG_TL 1100
#define AG_WDELAY 0
#define AG_WNT ""

namespace {

constexpr int NR = 8;
constexpr int KDIM = 4096, NLOC = 2048, MSH = 512;
constexpr unsigned SHARD_BYTES = MSH * KDIM;
constexpr unsigned W_BYTES = KDIM * NLOC;
constexpr unsigned OUT_BYTES = NR * MSH * NLOC * 4u;
constexpr int NWG = 256;
constexpr int C_PUSH = 4, C_PF = 2, C_SVC = C_PUSH + C_PF;
constexpr int NPUSH = C_PUSH * 8;
constexpr int NCOPY = C_PF * 8;
constexpr int CW = 80;
constexpr size_t WS_TMO = 0;
constexpr size_t WS_CALL = 4096;
constexpr size_t WS_FLAGS = 64 * 1024;
constexpr size_t WS_STAMPS = 256 * 1024;
constexpr size_t WS_COPY = 1u << 20;
constexpr size_t WS_STAGE = WS_COPY + 2 * (size_t)SHARD_BYTES;
constexpr size_t WS_NEED = WS_STAGE + 2 * NR * (size_t)SHARD_BYTES;
constexpr int NS = AG_NS;
constexpr int NA = AG_NA;
static_assert(NS % 16 == 0 && NS >= 32 && NS <= 112 && NA % 16 == 0 && NA >= 32 && NA <= NS, "push / copy split");
constexpr unsigned long long DEADLINE_TICKS = 50000000ull;

constexpr int KB_LDS = 128 * CW;
constexpr int WSLOTS = 3;
constexpr int WAVE_LDS = WSLOTS * KB_LDS;
constexpr int PRS = 336;
constexpr int PBUF = 16 * PRS;
constexpr int LDS_CTRL = 4 * WAVE_LDS;
constexpr int LDS_BYTES = LDS_CTRL + 64;

typedef float f32x4 __attribute__((ext_vector_type(4)));
typedef int i32x8 __attribute__((ext_vector_type(8)));
typedef int i32x2 __attribute__((ext_vector_type(2)));
typedef int i32x4 __attribute__((ext_vector_type(4)));
typedef unsigned u32x4 __attribute__((ext_vector_type(4)));
#define LAS __attribute__((address_space(3)))
#define GAS __attribute__((address_space(1)))
#define DI __device__ __forceinline__
#define GAP() __builtin_amdgcn_sched_barrier(0)
template <int... Is> using iseq = std::integer_sequence<int, Is...>;
template <int N> using make_iseq = std::make_integer_sequence<int, N>;

#define ACLOB "a0","a1","a2","a3","a4","a5","a6","a7","a8","a9","a10","a11","a12","a13","a14","a15","a16","a17","a18","a19","a20","a21","a22","a23","a24","a25","a26","a27","a28","a29","a30","a31","a32","a33","a34","a35","a36","a37","a38","a39","a40","a41","a42","a43","a44","a45","a46","a47","a48","a49","a50","a51","a52","a53","a54","a55","a56","a57","a58","a59","a60","a61","a62","a63","a64","a65","a66","a67","a68","a69","a70","a71","a72","a73","a74","a75","a76","a77","a78","a79","a80","a81","a82","a83","a84","a85","a86","a87","a88","a89","a90","a91","a92","a93","a94","a95","a96","a97","a98","a99","a100","a101","a102","a103","a104","a105","a106","a107","a108","a109","a110","a111","a112","a113","a114","a115","a116","a117","a118","a119","a120","a121","a122","a123","a124","a125","a126","a127", \
  "a128","a129","a130","a131","a132","a133","a134","a135","a136","a137","a138","a139","a140","a141","a142","a143","a144","a145","a146","a147","a148","a149","a150","a151","a152","a153","a154","a155","a156","a157","a158","a159","a160","a161","a162","a163","a164","a165","a166","a167","a168","a169","a170","a171","a172","a173","a174","a175","a176","a177","a178","a179","a180","a181","a182","a183","a184","a185","a186","a187","a188","a189","a190","a191","a192","a193","a194","a195","a196","a197","a198","a199","a200","a201","a202","a203","a204","a205","a206","a207","a208","a209","a210","a211","a212","a213","a214","a215","a216","a217","a218","a219","a220","a221","a222","a223","a224","a225","a226","a227","a228","a229","a230","a231","a232","a233","a234","a235","a236","a237","a238","a239","a240","a241","a242","a243","a244","a245","a246","a247","a248","a249","a250","a251","a252","a253","a254","a255"

struct Params {
  char* ws[NR];
  const unsigned char* x;
  const unsigned char* w;
  const float* sx;
  const float* sw;
  float* out;
  char* myws;
  int rank;
};

template <int N> DI void wait_vm() { asm volatile("s_waitcnt vmcnt(%0)" :: "n"(N) : "memory"); }
DI void wait_lgkm0() { asm volatile("s_waitcnt lgkmcnt(0)" ::: "memory"); }
DI unsigned long long rtc() { return __builtin_amdgcn_s_memrealtime(); }
DI unsigned rfl(unsigned v) { return __builtin_amdgcn_readfirstlane(v); }

struct St {
  __amdgpu_buffer_rsrc_t rx, rx2, rw, ro;
  unsigned vx;
  unsigned sx;
  unsigned vw[5];
  unsigned sw;
  int rot;
  unsigned ldsw;
  unsigned tr;
  unsigned pw;
  unsigned pr[2];
  unsigned vo[2];
  unsigned so;
  i32x8 wv[16];
  f32x4 acc[2][5];
  f32x4 pt[2][4];
  unsigned scale_bits;
};

DI void w_dma(St& s, int kb, int slot) {
#pragma unroll
  for (int i = 0; i < 10; ++i) {
    const unsigned m0v = s.ldsw + slot * KB_LDS + i * 1024;
    const unsigned so = s.sw + (unsigned)(((kb + s.rot) & 7) * 128 + (i / 5) * 64) * NLOC;
    asm volatile("s_mov_b32 m0, %0\n\ts_nop 0\n\tbuffer_load_dwordx4 %1, %2, %3 offen" AG_WNT " lds" :: "s"(m0v), "v"(s.vw[i % 5]), "s"(s.rw), "s"(so) : "memory");
  }
}
template <int F, int SLOT> DI void w_tr(St& s) {
  constexpr int O = SLOT * KB_LDS + (F % 5) * 16;
  if constexpr (F < 24) {
    constexpr int A = 64 + 8 * F;
    asm volatile("ds_read_b64_tr_b8 a[%c1:%c2], %0 offset:%c9\n\t"
                 "ds_read_b64_tr_b8 a[%c3:%c4], %0 offset:%c10\n\t"
                 "ds_read_b64_tr_b8 a[%c5:%c6], %0 offset:%c11\n\t"
                 "ds_read_b64_tr_b8 a[%c7:%c8], %0 offset:%c12"
                 :: "v"(s.tr), "i"(A), "i"(A + 1), "i"(A + 2), "i"(A + 3), "i"(A + 4), "i"(A + 5), "i"(A + 6), "i"(A + 7),
                    "i"(O), "i"(O + 8 * CW), "i"(O + 16 * CW), "i"(O + 24 * CW)
                 : "memory", ACLOB);
  } else {
    const unsigned base = s.tr + O;
    i32x2 r0 = __builtin_amdgcn_ds_read_tr8_b64_v2i32((LAS i32x2*)(base));
    i32x2 r1 = __builtin_amdgcn_ds_read_tr8_b64_v2i32((LAS i32x2*)(base + 8 * CW));
    i32x2 r2 = __builtin_amdgcn_ds_read_tr8_b64_v2i32((LAS i32x2*)(base + 16 * CW));
    i32x2 r3 = __builtin_amdgcn_ds_read_tr8_b64_v2i32((LAS i32x2*)(base + 24 * CW));
    i32x4 lo = __builtin_shufflevector(r0, r1, 0, 1, 2, 3), hi = __builtin_shufflevector(r2, r3, 0, 1, 2, 3);
    s.wv[F - 24] = __builtin_shufflevector(lo, hi, 0, 1, 2, 3, 4, 5, 6, 7);
  }
}
template <int KB> DI void w_stage(St& s) {
  constexpr int LEFT = (7 - KB) < (WSLOTS - 1) ? (7 - KB) : (WSLOTS - 1);
  wait_vm<10 * LEFT>();
  w_tr<5 * KB + 0, KB % WSLOTS>(s); w_tr<5 * KB + 1, KB % WSLOTS>(s); w_tr<5 * KB + 2, KB % WSLOTS>(s);
  w_tr<5 * KB + 3, KB % WSLOTS>(s); w_tr<5 * KB + 4, KB % WSLOTS>(s);
  GAP();
  wait_lgkm0();
  if constexpr (KB + WSLOTS < 8) w_dma(s, KB + WSLOTS, KB % WSLOTS);
  GAP();
}
template <int... Is> DI void w_stage_all(St& s, iseq<Is...>) { (w_stage<Is>(s), ...); }

template <int SLOT> DI void x_load(St& s) {
  constexpr int A = 8 * SLOT;
  asm volatile("buffer_load_dwordx4 a[%c3:%c4], %0, %1, %2 offen\n\t"
               "buffer_load_dwordx4 a[%c5:%c6], %0, %1, %2 offen offset:1024"
               :: "v"(s.vx), "s"(s.rx), "s"(s.sx), "i"(A), "i"(A + 3), "i"(A + 4), "i"(A + 7) : "memory", ACLOB);
  s.sx += 8192u;
}
template <int... Is> DI void x_fill(St& s, iseq<Is...>) { (x_load<Is>(s), ...); }
template <int SET, int KB, int U> DI void mma(St& s) {
  constexpr int X = 8 * KB, F = 5 * KB + U;
  if constexpr (F < 24) {
    constexpr int A = 64 + 8 * F;
    if constexpr (KB == 0) asm volatile("v_mfma_f32_16x16x128_f8f6f4 %0, a[%c1:%c2], a[%c3:%c4], 0" : "=v"(s.acc[SET][U]) : "i"(A), "i"(A + 7), "i"(X), "i"(X + 7) : ACLOB);
    else asm volatile("v_mfma_f32_16x16x128_f8f6f4 %0, a[%c1:%c2], a[%c3:%c4], %0" : "+v"(s.acc[SET][U]) : "i"(A), "i"(A + 7), "i"(X), "i"(X + 7) : ACLOB);
  } else {
    asm volatile("v_mfma_f32_16x16x128_f8f6f4 %0, %1, a[%c2:%c3], %0" : "+v"(s.acc[SET][U]) : "v"(s.wv[F - 24]), "i"(X), "i"(X + 7) : ACLOB);
  }
}
template <int SET, int U> DI void red_write(St& s) { *(LAS f32x4*)(s.pw + SET * PBUF + U * 64) = s.acc[SET][U]; }
DI void red_barrier() { wait_lgkm0(); __builtin_amdgcn_s_barrier(); asm volatile("" ::: "memory"); }
template <int SET, int T, int W> DI void red_read(St& s) { s.pt[T][W] = *(LAS f32x4*)(s.pr[T] + SET * PBUF + W * WAVE_LDS); }
DI float fadd(float a, float b) { float r; asm("v_add_f32_e32 %0, %1, %2" : "=v"(r) : "v"(a), "v"(b)); return r; }
template <int T, int A, int B> DI void red_add(St& s) {
#pragma unroll
  for (int e = 0; e < 4; ++e) s.pt[T][A][e] = fadd(s.pt[T][A][e], s.pt[T][B][e]);
}
template <int T> DI void red_scale(St& s) {
#pragma unroll
  for (int e = 0; e < 4; ++e) asm("v_mul_f32_e32 %0, %1, %2" : "=v"(s.pt[T][0][e]) : "s"(s.scale_bits), "v"(s.pt[T][0][e]));
}
template <int T> DI void red_store(St& s) {
  f32x4 y;
#pragma unroll
  for (int e = 0; e < 4; ++e) y[e] = __builtin_elementwise_maximum(s.pt[T][0][e], 0.f);
  __builtin_amdgcn_raw_buffer_store_b128(__builtin_bit_cast(u32x4, y), s.ro, s.vo[T], s.so, AG_OUT_AUX);
}
template <int P, int G> DI void finish(St& s) {
  if constexpr (G >= 2 && G <= 6) red_write<P, G - 2>(s);
  if constexpr (G == 11) red_barrier();
  if constexpr (G == 12) { red_read<P, 0, 0>(s); red_read<P, 0, 1>(s); }
  if constexpr (G == 13) { red_read<P, 0, 2>(s); red_read<P, 0, 3>(s); }
  if constexpr (G == 14) { red_read<P, 1, 0>(s); red_read<P, 1, 1>(s); }
  if constexpr (G == 16) { red_read<P, 1, 2>(s); red_read<P, 1, 3>(s); }
  if constexpr (G == 21) red_add<0, 0, 1>(s);
  if constexpr (G == 22) red_add<0, 2, 3>(s);
  if constexpr (G == 23) red_add<0, 0, 2>(s);
  if constexpr (G == 24) red_scale<0>(s);
  if constexpr (G == 26) red_store<0>(s);
  if constexpr (G == 27) red_add<1, 0, 1>(s);
  if constexpr (G == 28) red_add<1, 2, 3>(s);
  if constexpr (G == 29) red_add<1, 0, 2>(s);
  if constexpr (G == 31) red_scale<1>(s);
  if constexpr (G == 32) { red_store<1>(s); s.so += 16u * NLOC * 4u; }
}
template <int P, int... Gs> DI void finish_all(St& s, iseq<Gs...>) { (finish<P, Gs>(s), ...); }
template <int SET, bool FIN, int KB, int U> DI void gap(St& s) {
  mma<SET, KB, U>(s);
  if constexpr (U == 0) {
    if constexpr (KB == 8 - AG_D) { if (s.sx == NS * 8192u) s.rx = s.rx2; }
    x_load<(KB + AG_D) & 7>(s);
  }
  if constexpr (FIN) finish<SET ^ 1, 5 * KB + U>(s);
  GAP();
}
template <int SET, bool FIN, int KB> DI void step(St& s) {
  wait_vm<2 * (AG_D - 1)>();
  gap<SET, FIN, KB, 0>(s); gap<SET, FIN, KB, 1>(s); gap<SET, FIN, KB, 2>(s); gap<SET, FIN, KB, 3>(s); gap<SET, FIN, KB, 4>(s);
}
template <int SET, bool FIN, int... Is> DI void group(St& s, iseq<Is...>) { (step<SET, FIN, Is>(s), ...); }

DI bool wait_flags(char* myws, int src, unsigned stage, int npush, bool copy, unsigned call, int lane) {
  GAS unsigned* fl = (GAS unsigned*)(myws + WS_FLAGS) + src * 64 + lane;
  GAS unsigned* tmo = (GAS unsigned*)(myws + WS_TMO);
  const unsigned long long t0 = rtc();
  for (unsigned it = 0;; ++it) {
    const unsigned f = __hip_atomic_load(fl, __ATOMIC_RELAXED, __HIP_MEMORY_SCOPE_SYSTEM);
    const unsigned long long m = __ballot(f >= call * 8u + stage);
    if ((!copy || ((m >> NPUSH) & ((1ull << NCOPY) - 1)) == (1ull << NCOPY) - 1) && __builtin_popcount((unsigned)m) >= npush) return false;
    if ((it & 63u) == 63u) {
      if (rfl(__hip_atomic_load(tmo, __ATOMIC_RELAXED, __HIP_MEMORY_SCOPE_AGENT)) == call || rtc() - t0 > DEADLINE_TICKS) {
        if (lane == 0) __hip_atomic_store(tmo, call, __ATOMIC_RELAXED, __HIP_MEMORY_SCOPE_AGENT);
        return true;
      }
    }
    __builtin_amdgcn_s_sleep(1);
  }
}
static_assert(NPUSH == 32 && NPUSH + NCOPY <= 64, "flag word layout");
DI unsigned unit_src(int u, int rank, int lane) {
  const int pos = u >> 3, q = (u >> 1) & 3, h = u & 1;
  return (unsigned)(((pos >> 3) * 16 + (lane & 15)) * KDIM + q * 1024 + ((pos + rank) & 7) * 128 + (lane >> 4) * 32 + h * 16);
}

#define STAMP(i) do { } while (0)

__global__ void __launch_bounds__(256, 1) ag_gemm_kernel(Params p) {
  __shared__ __attribute__((aligned(1024))) unsigned char lds[LDS_BYTES];
  const int tid = threadIdx.x, lane = tid & 63;
  const int wave = __builtin_amdgcn_readfirstlane(tid >> 6);
  const int b = blockIdx.x, src = b & 7, c = b >> 3;
  const int rank = p.rank & 7;
  char* const myws = p.myws;
  GAS unsigned* const cnt = (GAS unsigned*)(myws + WS_CALL) + b * 16;
  STAMP(0);

  if (c < C_PUSH) {
    const int pw = c * 8 + src, w = pw * 4 + wave;
    constexpr int U = NS / 16;
    u32x4 v[U];
    const unsigned char* xs = p.x + unit_src(w, rank, lane);
#pragma unroll
    for (int i = 0; i < U; ++i) v[i] = *(const u32x4*)(xs + (size_t)i * 32 * KDIM);
    const unsigned cv = __hip_atomic_load(cnt, __ATOMIC_RELAXED, __HIP_MEMORY_SCOPE_AGENT);
    char* dst = p.ws[lane & 7];
    char* pb[NR];
#pragma unroll
    for (int i = 0; i < NR; ++i) pb[i] = p.ws[(w + i) & 7];
    const unsigned call = rfl(cv) + 1u;
    const size_t soff = WS_STAGE + (size_t)((call & 1u) * NR + rank) * SHARD_BYTES;
    GAS unsigned* fdst = (GAS unsigned*)(dst + WS_FLAGS) + rank * 64 + pw;
    asm volatile("" : "+v"(fdst));
    wait_vm<0>();
#pragma unroll
    for (int i = 0; i <= U; ++i) {
      if (i < U) {
#pragma unroll
        for (int d = 0; d < NR; ++d) {
          __amdgpu_buffer_rsrc_t rd = __builtin_amdgcn_make_buffer_rsrc(pb[d] + soff, 0, NS * 8192, 0x00020000);
          __builtin_amdgcn_raw_buffer_store_b128(v[i], rd, (unsigned)(lane * 16), (unsigned)((w + 128 * i) * 1024), 17  );
        }
        if (i == 0) { STAMP(1); continue; }
        wait_vm<NR>();
      } else {
        wait_vm<0>();
      }
      if (i == NA / 16 || i >= U - 1) {
        __builtin_amdgcn_s_barrier();
        if (tid < NR) __hip_atomic_store(fdst, call * 8u + (unsigned)i, __ATOMIC_RELAXED, __HIP_MEMORY_SCOPE_SYSTEM);
        if (i == NA / 16) STAMP(2);
      }
    }
    if (tid == 0) __hip_atomic_store(cnt, call, __ATOMIC_RELAXED, __HIP_MEMORY_SCOPE_AGENT);
    STAMP(3);
    return;
  }
  char* sws = p.ws[src];
  if (c < C_SVC) {
    const unsigned long long tstart = rtc();
    const int cw = (c - C_PUSH) * 8 + src, u0 = NS * 8 + (cw * 4 + wave) * 2;
    constexpr int U = 16 - NS / 16;
    u32x4 v[2 * U];
    const unsigned char* xs = p.x + unit_src(u0, rank, lane);
#pragma unroll
    for (int i = 0; i < U; ++i) { v[2 * i] = *(const u32x4*)(xs + (size_t)i * 32 * KDIM); v[2 * i + 1] = *(const u32x4*)(xs + (size_t)i * 32 * KDIM + 16); }
    const unsigned cv = __hip_atomic_load(cnt, __ATOMIC_RELAXED, __HIP_MEMORY_SCOPE_AGENT);
    char* dst = p.ws[lane & 7];
    const unsigned call = rfl(cv) + 1u;
    __amdgpu_buffer_rsrc_t rc = __builtin_amdgcn_make_buffer_rsrc(myws + WS_COPY + (size_t)(call & 1u) * SHARD_BYTES, 0, (int)SHARD_BYTES, 0x00020000);
#pragma unroll
    for (int i = 0; i < 2 * U; ++i) __builtin_amdgcn_raw_buffer_store_b128(v[i], rc, (unsigned)(lane * 16 + (i & 1) * 1024), (unsigned)((u0 + 128 * (i >> 1)) * 1024), 17);
    GAS unsigned* fdst = (GAS unsigned*)(dst + WS_FLAGS) + rank * 64 + NPUSH + cw;
    asm volatile("" : "+v"(fdst));
    wait_vm<0>();
    __builtin_amdgcn_s_barrier();
    if (tid < NR) __hip_atomic_store(fdst, call * 8u + 7u, __ATOMIC_RELAXED, __HIP_MEMORY_SCOPE_SYSTEM);
    if (tid == 0) __hip_atomic_store(cnt, call, __ATOMIC_RELAXED, __HIP_MEMORY_SCOPE_AGENT);
    STAMP(1);
    if (src == rank) return;
    __amdgpu_buffer_rsrc_t rs = __builtin_amdgcn_make_buffer_rsrc(sws + WS_COPY + (size_t)(call & 1u) * SHARD_BYTES, 0, (int)SHARD_BYTES, 0x00020000);
    const unsigned vo = (unsigned)(wave * 2048 + lane * 16);
    unsigned so = (unsigned)(NS + c - C_PUSH) * 8192u;
    while (rtc() - tstart < AG_TL) __builtin_amdgcn_s_sleep(2);
    if (wait_flags(myws, src, NS / 16 - 1, AG_TH, true, call, lane)) return;
    STAMP(2);
    for (int i = 0; i < (256 - NS) / C_PF; ++i) {
      wait_vm<6>();
      asm volatile("buffer_load_dwordx4 a[0:3], %0, %1, %2 offen\n\tbuffer_load_dwordx4 a[4:7], %0, %1, %2 offen offset:1024" :: "v"(vo), "s"(rs), "s"(so) : "memory", ACLOB);
      so += C_PF * 8192u;
    }
    wait_vm<0>();
    STAMP(3);
    return;
  }

  const int cb = c - C_SVC;
  const unsigned ldsbase = (unsigned)(size_t)(LAS unsigned char*)lds;
  St s;
  s.ldsw = ldsbase + wave * WAVE_LDS;
  s.rw = __builtin_amdgcn_make_buffer_rsrc((void*)p.w, 0, (int)W_BYTES, 0x00020000);
#pragma unroll
  for (int i = 0; i < 5; ++i) { const int e = i * 64 + lane; s.vw[i] = (unsigned)((e / 5) * NLOC + (e % 5) * 16); }
  s.sw = (unsigned)(wave * 1024 * NLOC + cb * CW);
  s.rot = src;
  for (int i = 0; i < AG_WDELAY; ++i) __builtin_amdgcn_s_sleep(1);
  for (int kb = 0; kb < WSLOTS; ++kb) w_dma(s, kb, kb);
  GAP();
  s.tr = s.ldsw + (unsigned)((32 * (lane >> 4) + ((lane & 15) >> 1)) * CW + (lane & 1) * 8);
  w_stage_all(s, make_iseq<8>{});
  STAMP(1);
  if (wave == 0) {
    const unsigned cv = __hip_atomic_load(cnt, __ATOMIC_RELAXED, __HIP_MEMORY_SCOPE_AGENT);
    const float sc = __builtin_nontemporal_load(p.sx + (lane >> 6)) * __builtin_nontemporal_load(p.sw + (lane >> 6));
    const unsigned call = rfl(cv) + 1u;
    const bool bad = wait_flags(myws, src, NA / 16, NPUSH, false, call, lane);
    if (lane == 0) {
      *(LAS unsigned*)(ldsbase + LDS_CTRL) = bad ? 0u : call;
      *(LAS unsigned*)(ldsbase + LDS_CTRL + 4) = bad ? 0x7fc00000u : __float_as_uint(sc);
      __hip_atomic_store(cnt, call, __ATOMIC_RELAXED, __HIP_MEMORY_SCOPE_AGENT);
    }
  }
  red_barrier();
  STAMP(2);
  const unsigned ctrl = rfl(*(LAS unsigned*)(ldsbase + LDS_CTRL));
  s.scale_bits = rfl(*(LAS unsigned*)(ldsbase + LDS_CTRL + 4));
  s.rx2 = __builtin_amdgcn_make_buffer_rsrc(sws + WS_COPY + (size_t)(ctrl & 1u) * SHARD_BYTES, 0, ctrl ? (int)SHARD_BYTES : 0, 0x00020000);
  s.rx = __builtin_amdgcn_make_buffer_rsrc(myws + WS_STAGE + (size_t)((ctrl & 1u) * NR + src) * SHARD_BYTES, 0, ctrl ? (int)SHARD_BYTES : 0, 0x00020000);
  s.vx = (unsigned)(wave * 2048 + lane * 16);
  s.sx = 0;
  asm volatile("s_nop 4" ::: "memory");
  x_fill(s, make_iseq<AG_D>{});
  GAP();
  s.pw = s.ldsw + (unsigned)((lane & 15) * PRS + (lane >> 4) * 16);
#pragma unroll
  for (int t = 0; t < 2; ++t) {
    const int u = t == 0 ? lane : 64 + (lane & 15);
    const int row = wave * 4 + u / 20, quad = u % 20;
    const bool ok = (t == 0 || lane < 16) && cb * CW + quad * 4 < NLOC;
    s.pr[t] = ldsbase + (unsigned)(row * PRS + quad * 16);
    s.vo[t] = ok ? (unsigned)(row * NLOC * 4 + quad * 16) : 0x7ff00000u;
  }
  s.ro = __builtin_amdgcn_make_buffer_rsrc((void*)p.out, 0, (int)OUT_BYTES, 0x00020000);
  s.so = (unsigned)(src * MSH) * (unsigned)(NLOC * 4) + (unsigned)(cb * CW * 4);
  asm volatile("s_nop 1" ::: "memory");
  GAP();
  group<0, false>(s, make_iseq<8>{});
  for (int it = 0; it < 15; ++it) {
    if (it == NA / 16 - 1 && ctrl) {
      if (wait_flags(myws, src, NS / 16, NPUSH, true, ctrl, lane)) {
        s.rx = s.rx2 = __builtin_amdgcn_make_buffer_rsrc(myws, 0, 0, 0x00020000);
        s.scale_bits = 0x7fc00000u;
      }
      GAP();
    }
    group<1, true>(s, make_iseq<8>{});
    group<0, true>(s, make_iseq<8>{});
  }
  group<1, true>(s, make_iseq<8>{});
  STAMP(10);
  asm volatile("s_nop 15" ::: "memory");
  GAP();
  finish_all<1>(s, make_iseq<40>{});
  STAMP(11);
}

}

extern "C" void kernel_launch(int rank, int nranks, ncclComm_t comm,
                              hipStream_t stream,
                              void* const* d_in, const int* in_sizes, int n_in,
                              void* d_out, int out_size,
                              const int* in_dtypes, int out_dtype,
                              void* const* d_out_all,
                              void* const* d_ws_all, size_t ws_bytes) {
  (void)comm; (void)in_dtypes; (void)out_dtype; (void)d_out_all;
  if (nranks != NR || (unsigned)rank >= (unsigned)NR || n_in < 4 || ws_bytes < WS_NEED ||
      in_sizes[0] != MSH * KDIM || in_sizes[1] != KDIM * NLOC || out_size != NR * MSH * NLOC) {
    fprintf(stderr, "kernel_launch: unexpected problem shape\n");
    return;
  }
  Params p;
  for (int r = 0; r < NR; ++r) p.ws[r] = (char*)d_ws_all[r];
  p.x = (const unsigned char*)d_in[0];
  p.w = (const unsigned char*)d_in[1];
  p.sx = (const float*)d_in[2];
  p.sw = (const float*)d_in[3];
  p.out = (float*)d_out;
  p.myws = p.ws[rank];
  p.rank = rank;
  hipLaunchKernelGGL(ag_gemm_kernel, dim3(NWG), dim3(256), 0, stream, p);
  HIP_CHECK(hipGetLastError());
}
```

```cpp
#include <hip/hip_runtime.h>
#include <rccl/rccl.h>
#include <cstdio>
#include <utility>

#define HIP_CHECK(expr) do { hipError_t _e = (expr); if (_e != hipSuccess) { \
    fprintf(stderr, "HIP error %s at %s:%d\n", hipGetErrorString(_e), __FILE__, __LINE__); \
    return; } } while (0)

#define AG_OUT_AUX 17
#define AG_D 5
#define AG_NS 48
#define AG_TD 3
#define AG_TH 0
#define AG_TL 850
#define AG_WDELAY 0
#define AG_WNT ""

namespace {

constexpr int NR = 8;
constexpr int KDIM = 4096, NLOC = 2048, MSH = 512;
constexpr unsigned SHARD_BYTES = MSH * KDIM;
constexpr unsigned W_BYTES = KDIM * NLOC;
constexpr unsigned OUT_BYTES = NR * MSH * NLOC * 4u;
constexpr int NWG = 256;
constexpr int C_PUSH = 4, C_PF = 2, C_SVC = C_PUSH + C_PF;
constexpr int NPUSH = C_PUSH * 8;
constexpr int NCOPY = C_PF * 8;
constexpr int CW = 80;
constexpr size_t WS_TMO = 0;
constexpr size_t WS_CALL = 4096;
constexpr size_t WS_FLAGS = 64 * 1024;
constexpr size_t WS_STAMPS = 256 * 1024;
constexpr size_t WS_COPY = 1u << 20;
constexpr size_t WS_STAGE = WS_COPY + 2 * (size_t)SHARD_BYTES;
constexpr size_t WS_NEED = WS_STAGE + 2 * NR * (size_t)SHARD_BYTES;
constexpr int NS = AG_NS;
constexpr int NSTG = NS / 16;
static_assert(NS % 16 == 0 && NS >= 32 && NS <= 96, "push / copy split");
constexpr unsigned long long DEADLINE_TICKS = 50000000ull;

constexpr int KB_LDS = 128 * CW;
constexpr int WSLOTS = 3;
constexpr int WAVE_LDS = WSLOTS * KB_LDS;
constexpr int PRS = 272;
constexpr int P4OFF = 16 * PRS;
constexpr int PBUF = 5632;
constexpr int LDS_BYTES = 4 * WAVE_LDS;

typedef float f32x4 __attribute__((ext_vector_type(4)));
typedef int i32x8 __attribute__((ext_vector_type(8)));
typedef int i32x2 __attribute__((ext_vector_type(2)));
typedef int i32x4 __attribute__((ext_vector_type(4)));
typedef unsigned u32x4 __attribute__((ext_vector_type(4)));
#define LAS __attribute__((address_space(3)))
#define GAS __attribute__((address_space(1)))
#define DI __device__ __forceinline__
#define GAP() __builtin_amdgcn_sched_barrier(0)
template <int... Is> using iseq = std::integer_sequence<int, Is...>;
template <int N> using make_iseq = std::make_integer_sequence<int, N>;

#define ACLOB "a0","a1","a2","a3","a4","a5","a6","a7","a8","a9","a10","a11","a12","a13","a14","a15","a16","a17","a18","a19","a20","a21","a22","a23","a24","a25","a26","a27","a28","a29","a30","a31","a32","a33","a34","a35","a36","a37","a38","a39","a40","a41","a42","a43","a44","a45","a46","a47","a48","a49","a50","a51","a52","a53","a54","a55","a56","a57","a58","a59","a60","a61","a62","a63","a64","a65","a66","a67","a68","a69","a70","a71","a72","a73","a74","a75","a76","a77","a78","a79","a80","a81","a82","a83","a84","a85","a86","a87","a88","a89","a90","a91","a92","a93","a94","a95","a96","a97","a98","a99","a100","a101","a102","a103","a104","a105","a106","a107","a108","a109","a110","a111","a112","a113","a114","a115","a116","a117","a118","a119","a120","a121","a122","a123","a124","a125","a126","a127", \
  "a128","a129","a130","a131","a132","a133","a134","a135","a136","a137","a138","a139","a140","a141","a142","a143","a144","a145","a146","a147","a148","a149","a150","a151","a152","a153","a154","a155","a156","a157","a158","a159","a160","a161","a162","a163","a164","a165","a166","a167","a168","a169","a170","a171","a172","a173","a174","a175","a176","a177","a178","a179","a180","a181","a182","a183","a184","a185","a186","a187","a188","a189","a190","a191","a192","a193","a194","a195","a196","a197","a198","a199","a200","a201","a202","a203","a204","a205","a206","a207","a208","a209","a210","a211","a212","a213","a214","a215","a216","a217","a218","a219","a220","a221","a222","a223","a224","a225","a226","a227","a228","a229","a230","a231","a232","a233","a234","a235","a236","a237","a238","a239","a240","a241","a242","a243","a244","a245","a246","a247","a248","a249","a250","a251","a252","a253","a254","a255"

#define HOTCLOB

struct Params {
  char* ws[NR];
  const unsigned char* x;
  const unsigned char* w;
  const float* sx;
  const float* sw;
  float* out;
  char* myws;
  int rank;
};

template <int N> DI void wait_vm() { asm volatile("s_waitcnt vmcnt(%0)" :: "n"(N) : "memory"); }
DI void wait_lgkm0() { asm volatile("s_waitcnt lgkmcnt(0)" ::: "memory"); }
DI unsigned long long rtc() { return __builtin_amdgcn_s_memrealtime(); }
DI unsigned rfl(unsigned v) { return __builtin_amdgcn_readfirstlane(v); }

struct St {
  __amdgpu_buffer_rsrc_t rx, rx2, rw, ro;
  unsigned vx;
  unsigned sx;
  unsigned vw[5];
  unsigned sw;
  int rot;
  unsigned vt, st;
  unsigned ldsw;
  unsigned tr;
  unsigned pw, pw4;
  unsigned pr, prb, pr4, pr4b;
  unsigned vo[2];
  unsigned so;
  i32x8 wv[16];
  f32x4 acc[2][5];
  f32x4 pt[4];
  float p4[4];
  unsigned scale_bits;
  __amdgpu_buffer_rsrc_t rf;
  unsigned vf, fv;
};

DI void w_dma(St& s, int kb, int slot) {
#pragma unroll
  for (int i = 0; i < 10; ++i) {
    const unsigned m0v = s.ldsw + slot * KB_LDS + i * 1024;
    const unsigned so = s.sw + (unsigned)(((kb + s.rot) & 7) * 128 + (i / 5) * 64) * NLOC;
    asm volatile("s_mov_b32 m0, %0\n\ts_nop 0\n\tbuffer_load_dwordx4 %1, %2, %3 offen" AG_WNT " lds" :: "s"(m0v), "v"(s.vw[i % 5]), "s"(s.rw), "s"(so) : "memory");
  }
}
DI void w_touch(St& s, int kb) {
  const unsigned so = s.st + (unsigned)(((kb + s.rot) & 7) * 128) * NLOC;
  asm volatile("buffer_load_dword a0, %0, %1, %2 offen" :: "v"(s.vt), "s"(s.rw), "s"(so) : "memory", "a0");
}
template <int F, int SLOT> DI void w_tr(St& s) {
  constexpr int O = SLOT * KB_LDS + (F % 5) * 16;
  if constexpr (F < 24) {
    constexpr int A = 64 + 8 * F;
    asm volatile("ds_read_b64_tr_b8 a[%c1:%c2], %0 offset:%c9\n\t"
                 "ds_read_b64_tr_b8 a[%c3:%c4], %0 offset:%c10\n\t"
                 "ds_read_b64_tr_b8 a[%c5:%c6], %0 offset:%c11\n\t"
                 "ds_read_b64_tr_b8 a[%c7:%c8], %0 offset:%c12"
                 :: "v"(s.tr), "i"(A), "i"(A + 1), "i"(A + 2), "i"(A + 3), "i"(A + 4), "i"(A + 5), "i"(A + 6), "i"(A + 7),
                    "i"(O), "i"(O + 8 * CW), "i"(O + 16 * CW), "i"(O + 24 * CW)
                 : "memory", ACLOB);
  } else {
    const unsigned base = s.tr + O;
    i32x2 r0 = __builtin_amdgcn_ds_read_tr8_b64_v2i32((LAS i32x2*)(base));
    i32x2 r1 = __builtin_amdgcn_ds_read_tr8_b64_v2i32((LAS i32x2*)(base + 8 * CW));
    i32x2 r2 = __builtin_amdgcn_ds_read_tr8_b64_v2i32((LAS i32x2*)(base + 16 * CW));
    i32x2 r3 = __builtin_amdgcn_ds_read_tr8_b64_v2i32((LAS i32x2*)(base + 24 * CW));
    i32x4 lo = __builtin_shufflevector(r0, r1, 0, 1, 2, 3), hi = __builtin_shufflevector(r2, r3, 0, 1, 2, 3);
    s.wv[F - 24] = __builtin_shufflevector(lo, hi, 0, 1, 2, 3, 4, 5, 6, 7);
  }
}
#define AG_FPOL " sc1"
DI void f_sample(St& s) { asm volatile("buffer_load_dword %0, %1, %2, 0 offen" AG_FPOL : "=v"(s.fv) : "v"(s.vf), "s"(s.rf) : "memory"); }
template <int KB> DI void w_stage(St& s) {
  constexpr int LEFT = (7 - KB) < (WSLOTS - 1) ? (7 - KB) : (WSLOTS - 1);
  wait_vm<10 * LEFT>();
  if constexpr (KB + AG_TD < 8) w_touch(s, KB + AG_TD);
  w_tr<5 * KB + 0, KB % WSLOTS>(s); w_tr<5 * KB + 1, KB % WSLOTS>(s); w_tr<5 * KB + 2, KB % WSLOTS>(s);
  w_tr<5 * KB + 3, KB % WSLOTS>(s); w_tr<5 * KB + 4, KB % WSLOTS>(s);
  GAP();
  wait_lgkm0();
  if constexpr (KB + WSLOTS == 7) f_sample(s);
  if constexpr (KB + WSLOTS < 8) w_dma(s, KB + WSLOTS, KB % WSLOTS);
  GAP();
}
template <int... Is> DI void w_stage_all(St& s, iseq<Is...>) { (w_stage<Is>(s), ...); }

template <int SLOT> DI void x_load(St& s) {
  constexpr int A = 8 * SLOT;
  asm volatile("buffer_load_dwordx4 a[%c3:%c4], %0, %1, %2 offen\n\t"
               "buffer_load_dwordx4 a[%c5:%c6], %0, %1, %2 offen offset:1024"
               :: "v"(s.vx), "s"(s.rx), "s"(s.sx), "i"(A), "i"(A + 3), "i"(A + 4), "i"(A + 7) : "memory", ACLOB);
  s.sx += 8192u;
}
template <int... Is> DI void x_fill(St& s, iseq<Is...>) { (x_load<Is>(s), ...); }
template <int SET, int KB, int U> DI void mma(St& s) {
  constexpr int X = 8 * KB, F = 5 * KB + U;
  if constexpr (F < 24) {
    constexpr int A = 64 + 8 * F;
    if constexpr (KB == 0) asm volatile("v_mfma_f32_16x16x128_f8f6f4 %0, a[%c1:%c2], a[%c3:%c4], 0" : "=v"(s.acc[SET][U]) : "i"(A), "i"(A + 7), "i"(X), "i"(X + 7) HOTCLOB);
    else asm volatile("v_mfma_f32_16x16x128_f8f6f4 %0, a[%c1:%c2], a[%c3:%c4], %0" : "+v"(s.acc[SET][U]) : "i"(A), "i"(A + 7), "i"(X), "i"(X + 7) HOTCLOB);
  } else {
    asm volatile("v_mfma_f32_16x16x128_f8f6f4 %0, %1, a[%c2:%c3], %0" : "+v"(s.acc[SET][U]) : "v"(s.wv[F - 24]), "i"(X), "i"(X + 7) HOTCLOB);
  }
}
DI void red_barrier() { wait_lgkm0(); __builtin_amdgcn_s_barrier(); asm volatile("" ::: "memory"); }
DI float fadd(float a, float b) { float r; asm("v_add_f32_e32 %0, %1, %2" : "=v"(r) : "v"(a), "v"(b)); return r; }
DI float fscale(unsigned sb, float a) { float r; asm("v_mul_f32_e32 %0, %1, %2" : "=v"(r) : "s"(sb), "v"(a)); return r; }
DI float frelu(float a) { return __builtin_elementwise_maximum(a, 0.f); }
template <int P, int V> DI void red_op(St& s) {
  constexpr int e = V & 3;
  if constexpr (V < 4) s.pt[0][e] = fadd(s.pt[0][e], s.pt[1][e]);
  else if constexpr (V < 8) s.pt[2][e] = fadd(s.pt[2][e], s.pt[3][e]);
  else if constexpr (V < 12) s.pt[0][e] = fadd(s.pt[0][e], s.pt[2][e]);
  else if constexpr (V < 16) s.pt[0][e] = fscale(s.scale_bits, s.pt[0][e]);
  else if constexpr (V < 20) s.pt[0][e] = frelu(s.pt[0][e]);
  else if constexpr (V == 20) s.p4[0] = fadd(s.p4[0], s.p4[1]);
  else if constexpr (V == 21) s.p4[2] = fadd(s.p4[2], s.p4[3]);
  else if constexpr (V == 22) s.p4[0] = fadd(s.p4[0], s.p4[2]);
  else if constexpr (V == 23) s.p4[0] = fscale(s.scale_bits, s.p4[0]);
  else if constexpr (V == 24) s.p4[0] = frelu(s.p4[0]);
}
template <int P, int G> DI void finish(St& s) {
  if constexpr (G >= 1 && G <= 4) *(LAS f32x4*)(s.pw + P * PBUF + (G - 1) * 64) = s.acc[P][G - 1];
  if constexpr (G == 6) *(LAS f32x4*)(s.pw4 + P * PBUF) = s.acc[P][4];
  if constexpr (G == 9) red_barrier();
  if constexpr (G == 11 || G == 12) s.pt[G - 11] = *(LAS f32x4*)(s.pr + P * PBUF + (G - 11) * WAVE_LDS);
  if constexpr (G == 13 || G == 14) s.pt[G - 11] = *(LAS f32x4*)(s.prb + P * PBUF + (G - 13) * WAVE_LDS);
  if constexpr (G == 16) { s.p4[0] = *(LAS float*)(s.pr4 + P * PBUF); s.p4[1] = *(LAS float*)(s.pr4 + P * PBUF + WAVE_LDS); }
  if constexpr (G == 17) { s.p4[2] = *(LAS float*)(s.pr4b + P * PBUF); s.p4[3] = *(LAS float*)(s.pr4b + P * PBUF + WAVE_LDS); }
  if constexpr (G >= 18 && G % 5 != 0) {
    constexpr int n = (G - 18) - (G / 5 - 3);
    if constexpr (n < 10) { red_op<P, 2 * n>(s); red_op<P, 2 * n + 1>(s); }
    if constexpr (n == 10) __builtin_amdgcn_raw_buffer_store_b128(__builtin_bit_cast(u32x4, s.pt[0]), s.ro, s.vo[0], s.so, AG_OUT_AUX);
    if constexpr (n == 11) { red_op<P, 20>(s); red_op<P, 21>(s); asm volatile("" :: "v"(s.pt[0])); }
    if constexpr (n >= 12 && n <= 14) red_op<P, n + 10>(s);
    if constexpr (n == 15) { __builtin_amdgcn_raw_buffer_store_b32(__float_as_uint(s.p4[0]), s.ro, s.vo[1], s.so, AG_OUT_AUX); s.so += 16u * NLOC * 4u; }
  }
}
template <int P, int... Gs> DI void finish_all(St& s, iseq<Gs...>) { (finish<P, Gs>(s), ...); }
template <int SET, int KB, int U> DI void gap(St& s) {
  mma<SET, KB, U>(s);
  if constexpr (U == 0) {
    if constexpr (SET == 1 && KB == 8 - AG_D) { if (s.sx == NS * 8192u) s.rx = s.rx2; }
    x_load<(KB + AG_D) & 7>(s);
  }
  if constexpr (SET == 0 && KB == 1 && U == 2) f_sample(s);
  finish<SET ^ 1, 5 * KB + U>(s);
  GAP();
}
static_assert(AG_D >= 2 && AG_D <= 6, "ring of 8 slots; the flag sample must retire inside its group");
template <int SET, int KB> DI void step(St& s) {
  wait_vm<2 * (AG_D - 1)>();
  gap<SET, KB, 0>(s); gap<SET, KB, 1>(s); gap<SET, KB, 2>(s); gap<SET, KB, 3>(s); gap<SET, KB, 4>(s);
}
template <int SET, int... Is> DI void group(St& s, iseq<Is...>) { (step<SET, Is>(s), ...); }

DI unsigned flags_known(unsigned f, unsigned call) {
  unsigned k = 0;
  while (k < NSTG && (unsigned)__ballot(f >= call * 8u + k + 1u) == 0xffffffffu) ++k;
  const bool cp = ((__ballot(f >= call * 8u + 7u) >> NPUSH) & ((1ull << NCOPY) - 1)) == (1ull << NCOPY) - 1;
  return (cp && k == NSTG) ? 99u : k;
}
DI unsigned wait_flags(char* myws, int src, unsigned stage, int npush, bool copy, unsigned call, int lane) {
  GAS unsigned* fl = (GAS unsigned*)(myws + WS_FLAGS) + src * 64 + lane;
  GAS unsigned* tmo = (GAS unsigned*)(myws + WS_TMO);
  const unsigned long long t0 = rtc();
  for (unsigned it = 0;; ++it) {
    const unsigned f = __hip_atomic_load(fl, __ATOMIC_RELAXED, __HIP_MEMORY_SCOPE_SYSTEM);
    const unsigned long long m = __ballot(f >= call * 8u + stage);
    const bool cp = ((m >> NPUSH) & ((1ull << NCOPY) - 1)) == (1ull << NCOPY) - 1;
    if ((!copy || cp) && __builtin_popcount((unsigned)m) >= npush) { const unsigned k = flags_known(f, call); return k ? k : 1u; }
    if ((it & 63u) == 63u) {
      if (rfl(__hip_atomic_load(tmo, __ATOMIC_RELAXED, __HIP_MEMORY_SCOPE_AGENT)) == call || rtc() - t0 > DEADLINE_TICKS) {
        if (lane == 0) __hip_atomic_store(tmo, call, __ATOMIC_RELAXED, __HIP_MEMORY_SCOPE_AGENT);
        return 0u;
      }
    }
    __builtin_amdgcn_s_sleep(1);
  }
}
DI int unit_of(int cb, int u) {
  if (cb < 16) return cb * 8 + u;
  const int l = (cb - 16) * 5 + u;
  return (l / 3) * 8 + 5 + l % 3;
}
static_assert(NPUSH == 32 && NPUSH + NCOPY <= 64, "flag word layout");
DI unsigned unit_src(int u, int rank, int lane) {
  const int pos = u >> 3, q = (u >> 1) & 3, h = u & 1;
  return (unsigned)(((pos >> 3) * 16 + (lane & 15)) * KDIM + q * 1024 + ((pos + rank) & 7) * 128 + (lane >> 4) * 32 + h * 16);
}

#define STAMP(i) do { } while (0)

__global__ void __launch_bounds__(256, 1) ag_gemm_kernel(Params p) {
  __shared__ __attribute__((aligned(1024))) unsigned char lds[LDS_BYTES];
  const int tid = threadIdx.x, lane = tid & 63;
  const int wave = __builtin_amdgcn_readfirstlane(tid >> 6);
  const int b = blockIdx.x, src = b & 7, c = b >> 3;
  const int rank = p.rank & 7;
  char* const myws = p.myws;
  GAS unsigned* const cnt = (GAS unsigned*)(myws + WS_CALL) + b * 16;
  STAMP(0);

  if (c < C_PUSH) {
    const int pw = c * 8 + src, w = pw * 4 + wave;
    constexpr int U = NS / 16;
    u32x4 v[U];
    const unsigned char* xs = p.x + unit_src(w, rank, lane);
#pragma unroll
    for (int i = 0; i < U; ++i) v[i] = *(const u32x4*)(xs + (size_t)i * 32 * KDIM);
    const unsigned cv = __hip_atomic_load(cnt, __ATOMIC_RELAXED, __HIP_MEMORY_SCOPE_AGENT);
    char* dst = p.ws[lane & 7];
    char* pb[NR];
#pragma unroll
    for (int i = 0; i < NR; ++i) pb[i] = p.ws[(w + i) & 7];
    const unsigned call = rfl(cv) + 1u;
    const size_t soff = WS_STAGE + (size_t)((call & 1u) * NR + rank) * SHARD_BYTES;
    GAS unsigned* fdst = (GAS unsigned*)(dst + WS_FLAGS) + rank * 64 + pw;
    asm volatile("" : "+v"(fdst));
    wait_vm<0>();
#pragma unroll
    for (int i = 0; i <= U; ++i) {
      if (i < U) {
#pragma unroll
        for (int d = 0; d < NR; ++d) {
          __amdgpu_buffer_rsrc_t rd = __builtin_amdgcn_make_buffer_rsrc(pb[d] + soff, 0, NS * 8192, 0x00020000);
          __builtin_amdgcn_raw_buffer_store_b128(v[i], rd, (unsigned)(lane * 16), (unsigned)((w + 128 * i) * 1024), 17  );
        }
        if (i == 0) { STAMP(1); continue; }
        wait_vm<NR>();
      } else {
        wait_vm<0>();
      }
      {
        __builtin_amdgcn_s_barrier();
        if (tid < NR) __hip_atomic_store(fdst, call * 8u + (unsigned)i, __ATOMIC_RELAXED, __HIP_MEMORY_SCOPE_SYSTEM);
        if (i == 2) STAMP(2);
      }
    }
    if (tid == 0) __hip_atomic_store(cnt, call, __ATOMIC_RELAXED, __HIP_MEMORY_SCOPE_AGENT);
    STAMP(3);
    return;
  }
  char* sws = p.ws[src];
  if (c < C_SVC) {
    const unsigned long long tstart = rtc();
    const int cw = (c - C_PUSH) * 8 + src, u0 = NS * 8 + (cw * 4 + wave) * 2;
    constexpr int U = 16 - NS / 16;
    u32x4 v[2 * U];
    const unsigned char* xs = p.x + unit_src(u0, rank, lane);
#pragma unroll
    for (int i = 0; i < U; ++i) { v[2 * i] = *(const u32x4*)(xs + (size_t)i * 32 * KDIM); v[2 * i + 1] = *(const u32x4*)(xs + (size_t)i * 32 * KDIM + 16); }
    const unsigned cv = __hip_atomic_load(cnt, __ATOMIC_RELAXED, __HIP_MEMORY_SCOPE_AGENT);
    char* dst = p.ws[lane & 7];
    const unsigned call = rfl(cv) + 1u;
    __amdgpu_buffer_rsrc_t rc = __builtin_amdgcn_make_buffer_rsrc(myws + WS_COPY + (size_t)(call & 1u) * SHARD_BYTES, 0, (int)SHARD_BYTES, 0x00020000);
#pragma unroll
    for (int i = 0; i < 2 * U; ++i) __builtin_amdgcn_raw_buffer_store_b128(v[i], rc, (unsigned)(lane * 16 + (i & 1) * 1024), (unsigned)((u0 + 128 * (i >> 1)) * 1024), 17);
    GAS unsigned* fdst = (GAS unsigned*)(dst + WS_FLAGS) + rank * 64 + NPUSH + cw;
    asm volatile("" : "+v"(fdst));
    wait_vm<0>();
    __builtin_amdgcn_s_barrier();
    if (tid < NR) __hip_atomic_store(fdst, call * 8u + 7u, __ATOMIC_RELAXED, __HIP_MEMORY_SCOPE_SYSTEM);
    if (tid == 0) __hip_atomic_store(cnt, call, __ATOMIC_RELAXED, __HIP_MEMORY_SCOPE_AGENT);
    STAMP(1);
    __amdgpu_buffer_rsrc_t rs = __builtin_amdgcn_make_buffer_rsrc(sws + WS_COPY + (size_t)(call & 1u) * SHARD_BYTES, 0, (int)SHARD_BYTES, 0x00020000);
    const unsigned vo = (unsigned)(wave * 2048 + lane * 16);
    unsigned so = (unsigned)(NS + c - C_PUSH) * 8192u;
    if (!wait_flags(myws, src, NSTG - 1, AG_TH, true, call, lane)) return;
    while (rtc() - tstart < AG_TL) __builtin_amdgcn_s_sleep(1);
    STAMP(2);
    for (int i = 0; i < (256 - NS) / C_PF; ++i) {
      wait_vm<6>();
      asm volatile("buffer_load_dwordx4 a[0:3], %0, %1, %2 offen\n\tbuffer_load_dwordx4 a[4:7], %0, %1, %2 offen offset:1024" :: "v"(vo), "s"(rs), "s"(so) : "memory", ACLOB);
      so += C_PF * 8192u;
    }
    wait_vm<0>();
    STAMP(3);
    return;
  }

  const int cb = c - C_SVC;
  const unsigned ldsbase = (unsigned)(size_t)(LAS unsigned char*)lds;
  St s;
  unsigned cv, sxv, swv;
  {
    const unsigned zero = 0;
    asm volatile("global_load_dword %0, %3, %4 sc1\n\tglobal_load_dword %1, %3, %5\n\tglobal_load_dword %2, %3, %6"
                 : "=&v"(cv), "=&v"(sxv), "=&v"(swv) : "v"(zero), "s"(cnt), "s"(p.sx), "s"(p.sw) : "memory");
  }
  s.ldsw = ldsbase + wave * WAVE_LDS;
  s.rw = __builtin_amdgcn_make_buffer_rsrc((void*)p.w, 0, (int)W_BYTES, 0x00020000);
#pragma unroll
  for (int i = 0; i < 5; ++i) {
    const int e = i * 64 + lane;
    s.vw[i] = (unsigned)((e / 5) * NLOC + (unit_of(cb, e % 5) & 127) * 16);
  }
  s.sw = (unsigned)(wave * 1024 * NLOC);
  s.vt = (unsigned)(lane * NLOC);
  {
    const unsigned l3 = (3u - (unsigned)((size_t)p.w >> 7)) & 7u;
    s.st = (cb < 16 && wave == (cb & 3)) ? (unsigned)((wave * 1024 + (cb >> 3) * 64) * NLOC) + (l3 + ((cb >> 2) & 1) * 8u) * 128u : 0x40000000u;
  }
  s.rot = src;
  for (int i = 0; i < AG_WDELAY; ++i) __builtin_amdgcn_s_sleep(1);
  for (int kb = 1; kb < AG_TD; ++kb) w_touch(s, kb);
  for (int kb = 0; kb < WSLOTS; ++kb) w_dma(s, kb, kb);
  GAP();
  s.tr = s.ldsw + (unsigned)((32 * (lane >> 4) + ((lane & 15) >> 1)) * CW + (lane & 1) * 8);
  s.vx = (unsigned)(wave * 2048 + lane * 16);
  s.sx = 0;
  s.pw = s.ldsw + (unsigned)((lane & 15) * PRS + (lane >> 4) * 16);
  s.pw4 = s.ldsw + (unsigned)(P4OFF + lane * 16 + (lane >> 4) * 32);
  {
    const int row = wave * 4 + (lane >> 4), col = lane & 15;
    s.pr = ldsbase + (unsigned)(row * PRS + col * 16);
    s.prb = s.pr + 2 * WAVE_LDS;
    s.pr4 = ldsbase + (unsigned)(P4OFF + row * 16 + (col >> 2) * (256 + 32) + (col & 3) * 4);
    s.pr4b = s.pr4 + 2 * WAVE_LDS;
    asm volatile("" : "+v"(s.pr), "+v"(s.prb), "+v"(s.pr4), "+v"(s.pr4b), "+v"(s.pw), "+v"(s.pw4));
    const int u0 = unit_of(cb, col >> 2), u4 = unit_of(cb, 4);
    s.vo[0] = u0 < 128 ? (unsigned)(row * NLOC * 4 + u0 * 64 + (col & 3) * 16) : 0x7ff00000u;
    s.vo[1] = u4 < 128 ? (unsigned)(row * NLOC * 4 + u4 * 64 + col * 4) : 0x7ff00000u;
  }
  const __amdgpu_buffer_rsrc_t ro = __builtin_amdgcn_make_buffer_rsrc((void*)p.out, 0, (int)OUT_BYTES, 0x00020000);
  s.ro = __builtin_amdgcn_make_buffer_rsrc((void*)p.out, 0, 0, 0x00020000);
  s.so = (unsigned)(src * MSH - 16) * (unsigned)(NLOC * 4);
  s.rf = __builtin_amdgcn_make_buffer_rsrc(myws + WS_FLAGS + src * 256, 0, 256, 0x00020000);
  s.vf = (unsigned)(lane * 4);
#pragma unroll
  for (int u = 0; u < 5; ++u) s.acc[1][u] = f32x4{0.f, 0.f, 0.f, 0.f};
  w_stage_all(s, make_iseq<7>{});
  asm volatile("" : "+v"(cv), "+v"(sxv), "+v"(swv), "+v"(s.fv));
  const unsigned call = rfl(cv) + 1u;
  s.scale_bits = rfl(__float_as_uint(__uint_as_float(sxv) * __uint_as_float(swv)));
  s.rx2 = __builtin_amdgcn_make_buffer_rsrc(sws + WS_COPY + (size_t)(call & 1u) * SHARD_BYTES, 0, (int)SHARD_BYTES, 0x00020000);
  s.rx = __builtin_amdgcn_make_buffer_rsrc(myws + WS_STAGE + (size_t)((call & 1u) * NR + src) * SHARD_BYTES, 0, (int)SHARD_BYTES, 0x00020000);
  STAMP(1);
  unsigned known = 0;
  auto require = [&](unsigned need) {
    known = flags_known(s.fv, call);
    if (known < need) {
      const bool last = need > NSTG;
      STAMP(4 + need);
      known = wait_flags(myws, src, last ? NSTG : need, NPUSH, last, call, lane);
      if (!known) {
        s.rx = s.rx2 = __builtin_amdgcn_make_buffer_rsrc(myws, 0, 0, 0x00020000);
        s.scale_bits = 0x7fc00000u;
        known = 99u;
      }
    }
    if (known == 99u) s.rf = __builtin_amdgcn_make_buffer_rsrc(myws, 0, 0, 0x00020000);
    __builtin_amdgcn_s_waitcnt(0xC07F);
    asm volatile("s_nop 4" ::: "memory");
  };
  require(1u);
  x_fill(s, make_iseq<AG_D>{});
  GAP();
  wait_vm<2 * AG_D>();
  STAMP(2);
  w_tr<35, 7 % WSLOTS>(s); w_tr<36, 7 % WSLOTS>(s); w_tr<37, 7 % WSLOTS>(s); w_tr<38, 7 % WSLOTS>(s); w_tr<39, 7 % WSLOTS>(s);
  __builtin_amdgcn_s_waitcnt(0xC07F);
  asm volatile("s_nop 1" ::: "memory");
  GAP();
  for (unsigned t = 0; t < 16u; ++t) {
    group<0>(s, make_iseq<8>{});
    s.ro = ro;
    if (__builtin_expect(known < t + 2u, 0)) {
      asm volatile("" : "+v"(s.fv));
      require(t + 2u);
    }
    GAP();
    group<1>(s, make_iseq<8>{});
  }
  STAMP(10);
  asm volatile("s_nop 15" ::: "memory");
  GAP();
  finish_all<1>(s, make_iseq<40>{});
  if (tid == 0) __hip_atomic_store(cnt, call, __ATOMIC_RELAXED, __HIP_MEMORY_SCOPE_AGENT);
  STAMP(11);
}

}

extern "C" void kernel_launch(int rank, int nranks, ncclComm_t comm,
                              hipStream_t stream,
                              void* const* d_in, const int* in_sizes, int n_in,
                              void* d_out, int out_size,
                              const int* in_dtypes, int out_dtype,
                              void* const* d_out_all,
                              void* const* d_ws_all, size_t ws_bytes) {
  (void)comm; (void)in_dtypes; (void)out_dtype; (void)d_out_all;
  if (nranks != NR || (unsigned)rank >= (unsigned)NR || n_in < 4 || ws_bytes < WS_NEED ||
      in_sizes[0] != MSH * KDIM || in_sizes[1] != KDIM * NLOC || out_size != NR * MSH * NLOC) {
    fprintf(stderr, "kernel_launch: unexpected problem shape\n");
    return;
  }
  Params p;
  for (int r = 0; r < NR; ++r) p.ws[r] = (char*)d_ws_all[r];
  p.x = (const unsigned char*)d_in[0];
  p.w = (const unsigned char*)d_in[1];
  p.sx = (const float*)d_in[2];
  p.sw = (const float*)d_in[3];
  p.out = (float*)d_out;
  p.myws = p.ws[rank];
  p.rank = rank;
  hipLaunchKernelGGL(ag_gemm_kernel, dim3(NWG), dim3(256), 0, stream, p);
  HIP_CHECK(hipGetLastError());
}
```

```cpp
#include <hip/hip_runtime.h>
#include <rccl/rccl.h>
#include <cstdio>
#include <utility>

#define HIP_CHECK(expr) do { hipError_t _e = (expr); if (_e != hipSuccess) { \
    fprintf(stderr, "HIP error %s at %s:%d\n", hipGetErrorString(_e), __FILE__, __LINE__); \
    return; } } while (0)

#define AG_DIAG 0
#define AG_SEQ
#define AG_EXACT
#define AG_OUT_AUX 17
#define AG_D 5
#define AG_NS 48
#define AG_PW 5
#define AG_TD 3
#define AG_TH 0
#define AG_TL 800
#define AG_TLOWN 1250
#define AG_WDELAY 0
#define AG_WNT ""

namespace {

constexpr int NR = 8;
constexpr int KDIM = 4096, NLOC = 2048, MSH = 512;
constexpr unsigned SHARD_BYTES = MSH * KDIM;
constexpr unsigned W_BYTES = KDIM * NLOC;
constexpr unsigned OUT_BYTES = NR * MSH * NLOC * 4u;
constexpr int NWG = 256;
constexpr int C_PUSH = 4, C_PF = 2, C_SVC = C_PUSH + C_PF;
constexpr int NPUSH = C_PUSH * 8;
constexpr int NCOPY = C_PF * 8;
constexpr int CW = 80;
constexpr size_t WS_TMO = 0;
constexpr size_t WS_CALL = 4096;
constexpr size_t WS_FLAGS = 64 * 1024;
constexpr size_t WS_STAMPS = 256 * 1024;
constexpr size_t WS_COPY = 1u << 20;
constexpr size_t WS_STAGE = WS_COPY + 2 * (size_t)SHARD_BYTES;
constexpr size_t WS_NEED = WS_STAGE + 2 * NR * (size_t)SHARD_BYTES;
constexpr int NS = AG_NS;
constexpr int NSTG = NS / 16;
static_assert(NS % 16 == 0 && NS >= 32 && NS <= 96, "push / copy split");
constexpr unsigned long long DEADLINE_TICKS = 50000000ull;

constexpr int KB_LDS = 128 * CW;
constexpr int WSLOTS = 3;
constexpr int WAVE_LDS = WSLOTS * KB_LDS;
constexpr int PRS = 272;
constexpr int P4OFF = 16 * PRS;
constexpr int PBUF = 5632;
constexpr int LDS_CTRL = 4 * WAVE_LDS;
constexpr int LDS_BYTES = LDS_CTRL + 64;

typedef float f32x4 __attribute__((ext_vector_type(4)));
typedef int i32x8 __attribute__((ext_vector_type(8)));
typedef int i32x2 __attribute__((ext_vector_type(2)));
typedef int i32x4 __attribute__((ext_vector_type(4)));
typedef unsigned u32x4 __attribute__((ext_vector_type(4)));
#define LAS __attribute__((address_space(3)))
#define GAS __attribute__((address_space(1)))
#define DI __device__ __forceinline__
#define GAP() __builtin_amdgcn_sched_barrier(0)
template <int... Is> using iseq = std::integer_sequence<int, Is...>;
template <int N> using make_iseq = std::make_integer_sequence<int, N>;

#define ACLOB "a0","a1","a2","a3","a4","a5","a6","a7","a8","a9","a10","a11","a12","a13","a14","a15","a16","a17","a18","a19","a20","a21","a22","a23","a24","a25","a26","a27","a28","a29","a30","a31","a32","a33","a34","a35","a36","a37","a38","a39","a40","a41","a42","a43","a44","a45","a46","a47","a48","a49","a50","a51","a52","a53","a54","a55","a56","a57","a58","a59","a60","a61","a62","a63","a64","a65","a66","a67","a68","a69","a70","a71","a72","a73","a74","a75","a76","a77","a78","a79","a80","a81","a82","a83","a84","a85","a86","a87","a88","a89","a90","a91","a92","a93","a94","a95","a96","a97","a98","a99","a100","a101","a102","a103","a104","a105","a106","a107","a108","a109","a110","a111","a112","a113","a114","a115","a116","a117","a118","a119","a120","a121","a122","a123","a124","a125","a126","a127", \
  "a128","a129","a130","a131","a132","a133","a134","a135","a136","a137","a138","a139","a140","a141","a142","a143","a144","a145","a146","a147","a148","a149","a150","a151","a152","a153","a154","a155","a156","a157","a158","a159","a160","a161","a162","a163","a164","a165","a166","a167","a168","a169","a170","a171","a172","a173","a174","a175","a176","a177","a178","a179","a180","a181","a182","a183","a184","a185","a186","a187","a188","a189","a190","a191","a192","a193","a194","a195","a196","a197","a198","a199","a200","a201","a202","a203","a204","a205","a206","a207","a208","a209","a210","a211","a212","a213","a214","a215","a216","a217","a218","a219","a220","a221","a222","a223","a224","a225","a226","a227","a228","a229","a230","a231","a232","a233","a234","a235","a236","a237","a238","a239","a240","a241","a242","a243","a244","a245","a246","a247","a248","a249","a250","a251","a252","a253","a254","a255"

#define HOTCLOB

struct Params {
  char* ws[NR];
  const unsigned char* x;
  const unsigned char* w;
  const float* sx;
  const float* sw;
  float* out;
  char* myws;
  int rank;
};

template <int N> DI void wait_vm() { asm volatile("s_waitcnt vmcnt(%0)" :: "n"(N) : "memory"); }
DI void wait_vm_dyn(int n) {
  switch (n) {
    case 0: wait_vm<0>(); break;
    case 8: wait_vm<8>(); break;
    default: wait_vm<0>(); break;
  }
}
DI void wait_lgkm0() { asm volatile("s_waitcnt lgkmcnt(0)" ::: "memory"); }
DI unsigned long long rtc() { return __builtin_amdgcn_s_memrealtime(); }
DI unsigned rfl(unsigned v) { return __builtin_amdgcn_readfirstlane(v); }

struct St {
  __amdgpu_buffer_rsrc_t rx, rx2, rw, ro;
  unsigned vx;
  unsigned sx;
  unsigned vw[5];
  unsigned sw;
  int rot;
  unsigned vt, st;
  unsigned ldsw;
  unsigned tr;
  unsigned pw, pw4;
  unsigned pr, prb, pr4, pr4b;
  unsigned vo[2];
  unsigned so;
  i32x8 wv[16];
  f32x4 acc[2][5];
  f32x4 pt[4];
  float p4[4];
  unsigned scale_bits;
  __amdgpu_buffer_rsrc_t rf;
  unsigned vf, fv;
  unsigned sqw, sqr, sqv, sqn;
};

DI void w_dma(St& s, int kb, int slot) {
#pragma unroll
  for (int i = 0; i < 10; ++i) {
    const unsigned m0v = s.ldsw + slot * KB_LDS + i * 1024;
    const unsigned so = s.sw + (unsigned)(((kb + s.rot) & 7) * 128 + (i / 5) * 64) * NLOC;
    asm volatile("s_mov_b32 m0, %0\n\ts_nop 0\n\tbuffer_load_dwordx4 %1, %2, %3 offen" AG_WNT " lds" :: "s"(m0v), "v"(s.vw[i % 5]), "s"(s.rw), "s"(so) : "memory");
  }
}
DI void w_touch(St& s, int kb) {
  const unsigned so = s.st + (unsigned)(((kb + s.rot) & 7) * 128) * NLOC;
  asm volatile("buffer_load_dword a0, %0, %1, %2 offen" :: "v"(s.vt), "s"(s.rw), "s"(so) : "memory", "a0");
}
template <int F, int SLOT> DI void w_tr(St& s) {
  constexpr int O = SLOT * KB_LDS + (F % 5) * 16;
  if constexpr (F < 24) {
    constexpr int A = 64 + 8 * F;
    asm volatile("ds_read_b64_tr_b8 a[%c1:%c2], %0 offset:%c9\n\t"
                 "ds_read_b64_tr_b8 a[%c3:%c4], %0 offset:%c10\n\t"
                 "ds_read_b64_tr_b8 a[%c5:%c6], %0 offset:%c11\n\t"
                 "ds_read_b64_tr_b8 a[%c7:%c8], %0 offset:%c12"
                 :: "v"(s.tr), "i"(A), "i"(A + 1), "i"(A + 2), "i"(A + 3), "i"(A + 4), "i"(A + 5), "i"(A + 6), "i"(A + 7),
                    "i"(O), "i"(O + 8 * CW), "i"(O + 16 * CW), "i"(O + 24 * CW)
                 : "memory", ACLOB);
  } else {
    const unsigned base = s.tr + O;
    i32x2 r0 = __builtin_amdgcn_ds_read_tr8_b64_v2i32((LAS i32x2*)(base));
    i32x2 r1 = __builtin_amdgcn_ds_read_tr8_b64_v2i32((LAS i32x2*)(base + 8 * CW));
    i32x2 r2 = __builtin_amdgcn_ds_read_tr8_b64_v2i32((LAS i32x2*)(base + 16 * CW));
    i32x2 r3 = __builtin_amdgcn_ds_read_tr8_b64_v2i32((LAS i32x2*)(base + 24 * CW));
    i32x4 lo = __builtin_shufflevector(r0, r1, 0, 1, 2, 3), hi = __builtin_shufflevector(r2, r3, 0, 1, 2, 3);
    s.wv[F - 24] = __builtin_shufflevector(lo, hi, 0, 1, 2, 3, 4, 5, 6, 7);
  }
}
#define AG_FPOL " sc1"
DI void f_sample(St& s) { asm volatile("buffer_load_dword %0, %1, %2, 0 offen" AG_FPOL : "=v"(s.fv) : "v"(s.vf), "s"(s.rf) : "memory"); }
template <int KB> DI void w_stage(St& s) {
  constexpr int LEFT = (7 - KB) < (WSLOTS - 1) ? (7 - KB) : (WSLOTS - 1);
  wait_vm<10 * LEFT>();
  if constexpr (KB + AG_TD < 8) w_touch(s, KB + AG_TD);
  w_tr<5 * KB + 0, KB % WSLOTS>(s); w_tr<5 * KB + 1, KB % WSLOTS>(s); w_tr<5 * KB + 2, KB % WSLOTS>(s);
  w_tr<5 * KB + 3, KB % WSLOTS>(s); w_tr<5 * KB + 4, KB % WSLOTS>(s);
  GAP();
  wait_lgkm0();
  if constexpr (KB + WSLOTS == 7) f_sample(s);
  if constexpr (KB + WSLOTS < 8) w_dma(s, KB + WSLOTS, KB % WSLOTS);
  GAP();
}
template <int... Is> DI void w_stage_all(St& s, iseq<Is...>) { (w_stage<Is>(s), ...); }

template <int SLOT> DI void x_load(St& s) {
  constexpr int A = 8 * SLOT;
  asm volatile("buffer_load_dwordx4 a[%c3:%c4], %0, %1, %2 offen\n\t"
               "buffer_load_dwordx4 a[%c5:%c6], %0, %1, %2 offen offset:1024"
               :: "v"(s.vx), "s"(s.rx), "s"(s.sx), "i"(A), "i"(A + 3), "i"(A + 4), "i"(A + 7) : "memory", ACLOB);
  s.sx += 8192u;
}
template <int... Is> DI void x_fill(St& s, iseq<Is...>) { (x_load<Is>(s), ...); }
template <int SET, int KB, int U> DI void mma(St& s) {
  constexpr int X = 8 * KB, F = 5 * KB + U;
  if constexpr (F < 24) {
    constexpr int A = 64 + 8 * F;
    if constexpr (KB == 0) asm volatile("v_mfma_f32_16x16x128_f8f6f4 %0, a[%c1:%c2], a[%c3:%c4], 0" : "=v"(s.acc[SET][U]) : "i"(A), "i"(A + 7), "i"(X), "i"(X + 7) HOTCLOB);
    else asm volatile("v_mfma_f32_16x16x128_f8f6f4 %0, a[%c1:%c2], a[%c3:%c4], %0" : "+v"(s.acc[SET][U]) : "i"(A), "i"(A + 7), "i"(X), "i"(X + 7) HOTCLOB);
  } else {
    asm volatile("v_mfma_f32_16x16x128_f8f6f4 %0, %1, a[%c2:%c3], %0" : "+v"(s.acc[SET][U]) : "v"(s.wv[F - 24]), "i"(X), "i"(X + 7) HOTCLOB);
  }
}
DI void red_barrier() { wait_lgkm0(); __builtin_amdgcn_s_barrier(); asm volatile("" ::: "memory"); }
DI float fadd(float a, float b) { float r; asm("v_add_f32_e32 %0, %1, %2" : "=v"(r) : "v"(a), "v"(b)); return r; }
DI float fscale(unsigned sb, float a) { float r; asm("v_mul_f32_e32 %0, %1, %2" : "=v"(r) : "s"(sb), "v"(a)); return r; }
DI float frelu(float a) { return __builtin_elementwise_maximum(a, 0.f); }
template <int P, int V> DI void red_op(St& s) {
  constexpr int e = V & 3;
  if constexpr (V < 4) s.pt[0][e] = fadd(s.pt[0][e], s.pt[1][e]);
  else if constexpr (V < 8) s.pt[2][e] = fadd(s.pt[2][e], s.pt[3][e]);
  else if constexpr (V < 12) s.pt[0][e] = fadd(s.pt[0][e], s.pt[2][e]);
  else if constexpr (V < 16) s.pt[0][e] = fscale(s.scale_bits, s.pt[0][e]);
  else if constexpr (V < 20) s.pt[0][e] = frelu(s.pt[0][e]);
  else if constexpr (V == 20) s.p4[0] = fadd(s.p4[0], s.p4[1]);
  else if constexpr (V == 21) s.p4[2] = fadd(s.p4[2], s.p4[3]);
  else if constexpr (V == 22) s.p4[0] = fadd(s.p4[0], s.p4[2]);
  else if constexpr (V == 23) s.p4[0] = fscale(s.scale_bits, s.p4[0]);
  else if constexpr (V == 24) s.p4[0] = frelu(s.p4[0]);
}
#define AG_SCHED 2
struct Sched { int w[5], bar, r[4], r4[2], v0; };
constexpr Sched SCHEDS[] = {
  {{1, 2, 3, 4, 6}, 9, {11, 12, 13, 14}, {16, 17}, 18},
  {{1, 3, 6, 8, 11}, 14, {16, 17, 18, 19}, {21, 21}, 22},
  {{1, 2, 3, 4, 6}, 12, {13, 14, 16, 17}, {18, 19}, 21},
  {{1, 3, 6, 8, 9}, 12, {13, 14, 16, 17}, {18, 19}, 21},
  {{2, 4, 7, 9, 12}, 14, {16, 17, 18, 19}, {21, 21}, 22},
};
constexpr Sched SC = SCHEDS[AG_SCHED];
constexpr int vslot(int G) { int n = 0; for (int g = SC.v0; g < G; ++g) n += (g % 5 != 0); return n; }
template <int P, int G, bool TAIL = false> DI void finish(St& s) {
  if constexpr (G == SC.w[0]) *(LAS f32x4*)(s.pw + P * PBUF) = s.acc[P][0];
  if constexpr (G == SC.w[1]) *(LAS f32x4*)(s.pw + P * PBUF + 64) = s.acc[P][1];
  if constexpr (G == SC.w[2]) *(LAS f32x4*)(s.pw + P * PBUF + 128) = s.acc[P][2];
  if constexpr (G == SC.w[3]) *(LAS f32x4*)(s.pw + P * PBUF + 192) = s.acc[P][3];
  if constexpr (G == SC.w[4]) *(LAS f32x4*)(s.pw4 + P * PBUF) = s.acc[P][4];
  if constexpr (TAIL && G == SC.bar) red_barrier();
  if constexpr (!TAIL && G == SC.w[4] + 1) { asm("v_add_u32_e32 %0, 1, %1" : "=v"(s.sqn) : "v"(s.sqn)); *(volatile LAS unsigned*)(s.sqw) = s.sqn; }
  if constexpr (!TAIL && G == SC.bar) { s.sqv = *(volatile LAS unsigned*)(s.sqr); asm volatile("" ::: "memory"); }
  if constexpr (!TAIL && G == SC.v0) {
    if (__builtin_expect(__builtin_amdgcn_ballot_w64(s.sqv < s.sqn) != 0, 0)) {
      do { s.sqv = *(volatile LAS unsigned*)(s.sqr); } while (__builtin_amdgcn_ballot_w64(s.sqv < s.sqn) != 0);
      asm volatile("" ::: "memory");
      s.pt[0] = *(LAS f32x4*)(s.pr + P * PBUF); s.pt[1] = *(LAS f32x4*)(s.pr + P * PBUF + WAVE_LDS);
      s.pt[2] = *(LAS f32x4*)(s.prb + P * PBUF); s.pt[3] = *(LAS f32x4*)(s.prb + P * PBUF + WAVE_LDS);
      s.p4[0] = *(LAS float*)(s.pr4 + P * PBUF); s.p4[1] = *(LAS float*)(s.pr4 + P * PBUF + WAVE_LDS);
      s.p4[2] = *(LAS float*)(s.pr4b + P * PBUF); s.p4[3] = *(LAS float*)(s.pr4b + P * PBUF + WAVE_LDS);
    }
  }
  if constexpr (G == SC.r[0]) s.pt[0] = *(LAS f32x4*)(s.pr + P * PBUF);
  if constexpr (G == SC.r[1]) s.pt[1] = *(LAS f32x4*)(s.pr + P * PBUF + WAVE_LDS);
  if constexpr (G == SC.r[2]) s.pt[2] = *(LAS f32x4*)(s.prb + P * PBUF);
  if constexpr (G == SC.r[3]) s.pt[3] = *(LAS f32x4*)(s.prb + P * PBUF + WAVE_LDS);
  if constexpr (G == SC.r4[0]) { s.p4[0] = *(LAS float*)(s.pr4 + P * PBUF); s.p4[1] = *(LAS float*)(s.pr4 + P * PBUF + WAVE_LDS); }
  if constexpr (G == SC.r4[1]) { s.p4[2] = *(LAS float*)(s.pr4b + P * PBUF); s.p4[3] = *(LAS float*)(s.pr4b + P * PBUF + WAVE_LDS); }
  if constexpr (G >= SC.v0 && G % 5 != 0) {
    constexpr int n = vslot(G);
    if constexpr (n < 10) { red_op<P, 2 * n>(s); red_op<P, 2 * n + 1>(s); }
    if constexpr (n == 10) { __builtin_amdgcn_raw_buffer_store_b128(__builtin_bit_cast(u32x4, s.pt[0]), s.ro, s.vo[0], s.so, AG_OUT_AUX); red_op<P, 20>(s); }
    if constexpr (n == 11) { red_op<P, 21>(s); red_op<P, 22>(s); asm volatile("" :: "v"(s.pt[0])); }
    if constexpr (n == 12) { red_op<P, 23>(s); red_op<P, 24>(s); }
    if constexpr (n == 13) { __builtin_amdgcn_raw_buffer_store_b32(__float_as_uint(s.p4[0]), s.ro, s.vo[1], s.so, AG_OUT_AUX); s.so += 16u * NLOC * 4u; }
  }
}
static_assert(vslot(40) > 13, "the finish must fit one group");
template <int P, int... Gs> DI void finish_all(St& s, iseq<Gs...>) { (finish<P, Gs, true>(s), ...); }
#define AG_SAMPLE_KB (AG_D <= 5 ? 1 : 0)
template <int SET, int KB, int U> DI void gap(St& s) {
  mma<SET, KB, U>(s);
  if constexpr (U == 0) {
    if constexpr (SET == 1 && KB == 8 - AG_D) { if (s.sx == NS * 8192u) s.rx = s.rx2; }
    x_load<(KB + AG_D) & 7>(s);
  }
  if constexpr (SET == 0 && KB == AG_SAMPLE_KB && U == 2) f_sample(s);
  finish<SET ^ 1, 5 * KB + U>(s);
  GAP();
}
static_assert(AG_D >= 2 && AG_D <= 5 + (AG_SAMPLE_KB == 0), "ring of 8 slots; the flag sample must retire inside its group");
constexpr int gap_of(int n) { for (int g = SC.v0; g < 40; ++g) if (g % 5 != 0 && vslot(g) == n) return g; return -1; }
constexpr int younger_other(int set, int kb) {
  int c = 0;
  for (int p = 5 * (kb - AG_D); p < 5 * kb; ++p) {
    const int g = p < 0 ? p + 40 : p, st = p < 0 ? set ^ 1 : set;
    c += (g == gap_of(10)) + (g == gap_of(13)) + (st == 0 && g == 5 * AG_SAMPLE_KB + 2);
  }
  return c;
}
template <int SET, int KB> DI void step(St& s) {
  wait_vm<2 * (AG_D - 1) + younger_other(SET, KB)>();
  gap<SET, KB, 0>(s); gap<SET, KB, 1>(s); gap<SET, KB, 2>(s); gap<SET, KB, 3>(s); gap<SET, KB, 4>(s);
}
template <int SET, int... Is> DI void group(St& s, iseq<Is...>) { (step<SET, Is>(s), ...); }

DI unsigned flags_known(unsigned f, unsigned call) {
  unsigned k = 0;
  while (k < NSTG && (unsigned)__ballot(f >= call * 8u + k + 1u) == 0xffffffffu) ++k;
  const bool cp = ((__ballot(f >= call * 8u + 7u) >> NPUSH) & ((1ull << NCOPY) - 1)) == (1ull << NCOPY) - 1;
  return (cp && k == NSTG) ? 99u : k;
}
DI unsigned wait_flags(char* myws, int src, unsigned stage, int npush, bool copy, unsigned call, int lane) {
  GAS unsigned* fl = (GAS unsigned*)(myws + WS_FLAGS) + src * 64 + lane;
  GAS unsigned* tmo = (GAS unsigned*)(myws + WS_TMO);
  const unsigned long long t0 = rtc();
  for (unsigned it = 0;; ++it) {
    const unsigned f = __hip_atomic_load(fl, __ATOMIC_RELAXED, __HIP_MEMORY_SCOPE_SYSTEM);
    const unsigned long long m = __ballot(f >= call * 8u + stage);
    const bool cp = ((m >> NPUSH) & ((1ull << NCOPY) - 1)) == (1ull << NCOPY) - 1;
    if ((!copy || cp) && __builtin_popcount((unsigned)m) >= npush) { const unsigned k = flags_known(f, call); return k ? k : 1u; }
    if ((it & 63u) == 63u) {
      if (rfl(__hip_atomic_load(tmo, __ATOMIC_RELAXED, __HIP_MEMORY_SCOPE_AGENT)) == call || rtc() - t0 > DEADLINE_TICKS) {
        if (lane == 0) __hip_atomic_store(tmo, call, __ATOMIC_RELAXED, __HIP_MEMORY_SCOPE_AGENT);
        return 0u;
      }
    }
    __builtin_amdgcn_s_sleep(1);
  }
}
DI void unit_map(int cb, unsigned& base, unsigned& pack) {
  const unsigned l0 = (unsigned)(cb - 16) * 5u, q = (l0 * 43u) >> 7, r = l0 - 3u * q;
  base = cb < 16 ? (unsigned)cb * 8u : q * 8u + 5u + r;
  pack = cb < 16 ? 0x43210u : r == 0 ? 0x98210u : r == 1 ? 0x98710u : 0xe8760u;
}
static_assert(NPUSH == 32 && NPUSH + NCOPY <= 64, "flag word layout");
DI GAS unsigned* flag_dst(char* const (&pb)[NR], int word) {
  unsigned lo = 0, hi = 0;
#pragma unroll
  for (int i = 0; i < NR; ++i) {
    const size_t a = (size_t)pb[i] + WS_FLAGS + (size_t)word * 4;
    asm("v_writelane_b32 %0, %2, %c4\n\tv_writelane_b32 %1, %3, %c4" : "+v"(lo), "+v"(hi) : "s"((unsigned)a), "s"((unsigned)(a >> 32)), "i"(i));
  }
  return (GAS unsigned*)(((size_t)hi << 32) | lo);
}
DI unsigned unit_src(int u, int rank, int lane) {
  const int pos = u >> 3, q = (u >> 1) & 3, h = u & 1;
  return (unsigned)(((pos >> 3) * 16 + (lane & 15)) * KDIM + q * 1024 + ((pos + rank) & 7) * 128 + (lane >> 4) * 32 + h * 16);
}

#define STAMP(i) do { } while (0)

__global__ void __launch_bounds__(256, 1) ag_gemm_kernel(Params p) {
  __shared__ __attribute__((aligned(1024))) unsigned char lds[LDS_BYTES];
  const int tid = threadIdx.x, lane = tid & 63;
  const int wave = __builtin_amdgcn_readfirstlane(tid >> 6);
  const int b = blockIdx.x, src = b & 7, c = b >> 3;
  int rank = p.rank & 7;
  char* myws = p.myws;
  const unsigned char* xp = p.x;
  const unsigned char* wp = p.w;
  const float* sxp = p.sx;
  const float* swp = p.sw;
  char* pb[NR];
#pragma unroll
  for (int i = 0; i < NR; ++i) pb[i] = p.ws[(b * 4 + wave + i) & 7];
  asm volatile("" : "+s"(rank), "+s"(myws), "+s"(xp), "+s"(wp), "+s"(sxp), "+s"(swp), "+s"(pb[0]), "+s"(pb[1]), "+s"(pb[2]), "+s"(pb[3]), "+s"(pb[4]), "+s"(pb[5]), "+s"(pb[6]), "+s"(pb[7]));
  GAS unsigned* const cnt = (GAS unsigned*)(myws + WS_CALL) + b * 16;
  STAMP(0);

  if (__builtin_expect(c < C_PUSH, 1)) {
    const int pw = c * 8 + src, w = pw * 4 + wave;
    u32x4 v[NSTG];
    const GAS unsigned char* xs = (const GAS unsigned char*)xp + unit_src(w, rank, lane);
    v[0] = *(const GAS u32x4*)xs;
    const unsigned cv = __hip_atomic_load(cnt, __ATOMIC_RELAXED, __HIP_MEMORY_SCOPE_AGENT);
#pragma unroll
    for (int i = 1; i < NSTG; ++i) v[i] = *(const GAS u32x4*)(xs + (size_t)i * 32 * KDIM);
    __amdgpu_buffer_rsrc_t rd[NR];
#pragma unroll
    for (int d = 0; d < NR; ++d) rd[d] = __builtin_amdgcn_make_buffer_rsrc(pb[d] + WS_STAGE + (size_t)rank * SHARD_BYTES, 0, AG_DIAG & 1 ? 0 : (int)(NR * SHARD_BYTES) + NS * 8192, 0x00020000);
    GAS unsigned* fdst = flag_dst(pb, rank * 64 + pw);
    GAP();
    const unsigned call = rfl(cv) + 1u;
    const unsigned so = (call & 1u) * (NR * SHARD_BYTES) + (unsigned)w * 1024u;
#pragma unroll
    for (int n = 0; n < NSTG * NR + AG_PW; ++n) {
      if (n >= NSTG * NR && n != NSTG * NR + AG_PW - 1 && (n < AG_PW || (n - AG_PW) % NR != NR - 1)) continue;
      if (n >= AG_PW) {
        if (n < NSTG * NR) wait_vm<AG_PW - 1>(); else wait_vm_dyn(NSTG * NR + AG_PW - 1 - n);
        if ((n - AG_PW) % NR == NR - 1) {
          __builtin_amdgcn_s_barrier();
          if (tid < NR) __hip_atomic_store(fdst, call * 8u + (unsigned)((n - AG_PW) / NR + 1), __ATOMIC_RELAXED, __HIP_MEMORY_SCOPE_SYSTEM);
          if ((n - AG_PW) / NR == 1) STAMP(2);
        }
      }
      if (n < NSTG * NR) __builtin_amdgcn_raw_buffer_store_b128(v[n / NR], rd[n % NR], (unsigned)(lane * 16), so + (unsigned)(n / NR) * 131072u, 17  );
      if (n == 0) STAMP(1);
    }
    if (tid == 0) __hip_atomic_store(cnt, call, __ATOMIC_RELAXED, __HIP_MEMORY_SCOPE_AGENT);
    STAMP(3);
    return;
  }
  char* sws = p.ws[src];
  if (c < C_SVC) {
    const unsigned long long tstart = rtc();
    const int cw = (c - C_PUSH) * 8 + src, u0 = NS * 8 + (cw * 4 + wave) * 2;
    constexpr int U = 16 - NS / 16;
    u32x4 v[2 * U];
    const GAS unsigned char* xs = (const GAS unsigned char*)xp + unit_src(u0, rank, lane);
    const unsigned cv = __hip_atomic_load(cnt, __ATOMIC_RELAXED, __HIP_MEMORY_SCOPE_AGENT);
#pragma unroll
    for (int i = 0; i < U; ++i) { v[2 * i] = *(const GAS u32x4*)(xs + (size_t)i * 32 * KDIM); v[2 * i + 1] = *(const GAS u32x4*)(xs + (size_t)i * 32 * KDIM + 16); }
    const unsigned call = rfl(cv) + 1u;
    __amdgpu_buffer_rsrc_t rc = __builtin_amdgcn_make_buffer_rsrc(myws + WS_COPY + (size_t)(call & 1u) * SHARD_BYTES, 0, AG_DIAG & 2 ? 0 : (int)SHARD_BYTES, 0x00020000);
#pragma unroll
    for (int i = 0; i < 2 * U; ++i) __builtin_amdgcn_raw_buffer_store_b128(v[i], rc, (unsigned)(lane * 16 + (i & 1) * 1024), (unsigned)((u0 + 128 * (i >> 1)) * 1024), 17);
    GAS unsigned* fdst = flag_dst(pb, rank * 64 + NPUSH + cw);
    wait_vm<0>();
    __builtin_amdgcn_s_barrier();
    if (tid < NR) __hip_atomic_store(fdst, call * 8u + 7u, __ATOMIC_RELAXED, __HIP_MEMORY_SCOPE_SYSTEM);
    if (tid == 0) __hip_atomic_store(cnt, call, __ATOMIC_RELAXED, __HIP_MEMORY_SCOPE_AGENT);
    STAMP(1);
    __amdgpu_buffer_rsrc_t rs = __builtin_amdgcn_make_buffer_rsrc(sws + WS_COPY + (size_t)(call & 1u) * SHARD_BYTES, 0, (int)SHARD_BYTES, 0x00020000);
    const unsigned vo = (unsigned)(wave * 2048 + lane * 16);
    unsigned so = (unsigned)(NS + c - C_PUSH) * 8192u;
    if (wave == 0) *(LAS unsigned*)lds = wait_flags(myws, src, NSTG - 1, AG_TH, true, call, lane);
    red_barrier();
    if (!rfl(*(LAS unsigned*)lds)) return;
    while (rtc() - tstart < (src == rank ? AG_TLOWN : AG_TL)) __builtin_amdgcn_s_sleep(1);
    STAMP(2);
    for (int i = 0; i < (256 - NS) / C_PF; ++i) {
      wait_vm<6>();
      asm volatile("buffer_load_dwordx4 a[0:3], %0, %1, %2 offen\n\tbuffer_load_dwordx4 a[4:7], %0, %1, %2 offen offset:1024" :: "v"(vo), "s"(rs), "s"(so) : "memory", ACLOB);
      so += C_PF * 8192u;
    }
    wait_vm<0>();
    STAMP(3);
    return;
  }

  const int cb = c - C_SVC;
  const unsigned ldsbase = (unsigned)(size_t)(LAS unsigned char*)lds;
  St s;
  unsigned cv, sxv, swv;
  {
    const unsigned zero = 0;
    asm volatile("global_load_dword %0, %3, %4 sc1\n\tglobal_load_dword %1, %3, %5\n\tglobal_load_dword %2, %3, %6"
                 : "=&v"(cv), "=&v"(sxv), "=&v"(swv) : "v"(zero), "s"(cnt), "s"(sxp), "s"(swp) : "memory");
  }
  s.ldsw = ldsbase + wave * WAVE_LDS;
  s.rw = __builtin_amdgcn_make_buffer_rsrc((void*)wp, 0, (int)W_BYTES, 0x00020000);
  unsigned ubase, upack;
  unit_map(cb, ubase, upack);
#pragma unroll
  for (int i = 0; i < 5; ++i) {
    const unsigned e = (unsigned)(i * 64 + lane), q = (e * 205u) >> 10, r = e - 5u * q;
    s.vw[i] = q * NLOC + ((ubase + ((upack >> (4u * r)) & 15u)) & 127u) * 16u;
  }
  s.sw = (unsigned)(wave * 1024 * NLOC);
  s.vt = (unsigned)(lane * NLOC);
  {
    const unsigned l3 = (3u - (unsigned)((size_t)wp >> 7)) & 7u;
    s.st = ((unsigned)((wave * 1024 + ((cb >> 3) & 1) * 64) * NLOC) + (l3 + ((cb >> 2) & 1) * 8u) * 128u) | ((cb < 16 && wave == (cb & 3)) ? 0u : 0x40000000u);
  }
  s.rot = src;
  for (int i = 0; i < AG_WDELAY; ++i) __builtin_amdgcn_s_sleep(1);
  for (int kb = 1; kb < AG_TD; ++kb) w_touch(s, kb);
  STAMP(3);
  for (int kb = 0; kb < WSLOTS; ++kb) w_dma(s, kb, kb);
  GAP();
  s.tr = s.ldsw + (unsigned)((32 * (lane >> 4) + ((lane & 15) >> 1)) * CW + (lane & 1) * 8);
  s.vx = (unsigned)(wave * 2048 + lane * 16);
  s.sx = 0;
  s.pw = s.ldsw + (unsigned)((lane & 15) * PRS + (lane >> 4) * 16);
  s.pw4 = s.ldsw + (unsigned)(P4OFF + lane * 16 + (lane >> 4) * 32);
  {
    const int row = wave * 4 + (lane >> 4), col = lane & 15;
    s.pr = ldsbase + (unsigned)(row * PRS + col * 16);
    s.prb = s.pr + 2 * WAVE_LDS;
    s.pr4 = ldsbase + (unsigned)(P4OFF + row * 16 + (col >> 2) * (256 + 32) + (col & 3) * 4);
    s.pr4b = s.pr4 + 2 * WAVE_LDS;
    asm volatile("" : "+v"(s.pr), "+v"(s.prb), "+v"(s.pr4), "+v"(s.pr4b), "+v"(s.pw), "+v"(s.pw4));
    const unsigned u0 = ubase + ((upack >> (4 * (col >> 2))) & 15u), u4 = ubase + (upack >> 16);
    s.vo[0] = u0 < 128 ? (unsigned)(row * NLOC * 4 + u0 * 64 + (col & 3) * 16) : 0x7ff00000u;
    s.vo[1] = u4 < 128 ? (unsigned)(row * NLOC * 4 + u4 * 64 + col * 4) : 0x7ff00000u;
  }
  const __amdgpu_buffer_rsrc_t ro = __builtin_amdgcn_make_buffer_rsrc((void*)p.out, 0, (int)OUT_BYTES, 0x00020000);
  s.ro = __builtin_amdgcn_make_buffer_rsrc((void*)p.out, 0, 0, 0x00020000);
  s.so = (unsigned)(src * MSH - 16) * (unsigned)(NLOC * 4);
  s.rf = __builtin_amdgcn_make_buffer_rsrc(myws + WS_FLAGS + src * 256, 0, wave == 0 ? 256 : 0, 0x00020000);
  s.vf = (unsigned)(lane * 4);
  s.sqw = ldsbase + LDS_CTRL + 16 + wave * 4;
  s.sqr = ldsbase + LDS_CTRL + 16 + (lane & 3) * 4;
  s.sqn = 0;
  asm volatile("" : "+v"(s.sqw), "+v"(s.sqr), "+v"(s.sqn));
  *(volatile LAS unsigned*)(s.sqw) = 0u;
#pragma unroll
  for (int u = 0; u < 5; ++u) s.acc[1][u] = f32x4{0.f, 0.f, 0.f, 0.f};
  w_stage_all(s, make_iseq<7>{});
  asm volatile("" : "+v"(cv), "+v"(sxv), "+v"(swv), "+v"(s.fv));
  const unsigned call = rfl(cv) + 1u;
  s.scale_bits = rfl(__float_as_uint(__uint_as_float(sxv) * __uint_as_float(swv)));
  s.rx2 = __builtin_amdgcn_make_buffer_rsrc(sws + WS_COPY + (size_t)(call & 1u) * SHARD_BYTES, 0, (int)SHARD_BYTES, 0x00020000);
  s.rx = __builtin_amdgcn_make_buffer_rsrc(myws + WS_STAGE + (size_t)((call & 1u) * NR + src) * SHARD_BYTES, 0, (int)SHARD_BYTES, 0x00020000);
  STAMP(1);
  unsigned known = 0;
  auto require = [&](unsigned need) {
    if (wave == 0) {
      unsigned k = flags_known(s.fv, call);
      if (k < need) {
        const bool last = need > NSTG;
        STAMP(4 + need);
        k = wait_flags(myws, src, last ? NSTG : need, NPUSH, last, call, lane);
      }
      *(LAS unsigned*)(ldsbase + LDS_CTRL) = k;
    }
    red_barrier();
    known = rfl(*(LAS unsigned*)(ldsbase + LDS_CTRL));
    if (!known) {
      s.rx = s.rx2 = __builtin_amdgcn_make_buffer_rsrc(myws, 0, 0, 0x00020000);
      s.scale_bits = 0x7fc00000u;
      known = 99u;
    }
    if (known == 99u) s.rf = __builtin_amdgcn_make_buffer_rsrc(myws, 0, 0, 0x00020000);
    __builtin_amdgcn_s_waitcnt(0xC07F);
    asm volatile("s_nop 4" ::: "memory");
  };
  require(1u);
  x_fill(s, make_iseq<AG_D>{});
  GAP();
  wait_vm<2 * AG_D>();
  STAMP(2);
  w_tr<35, 7 % WSLOTS>(s); w_tr<36, 7 % WSLOTS>(s); w_tr<37, 7 % WSLOTS>(s); w_tr<38, 7 % WSLOTS>(s); w_tr<39, 7 % WSLOTS>(s);
  wait_vm<0>();
  __builtin_amdgcn_s_waitcnt(0xC07F);
  asm volatile("s_nop 1" ::: "memory");
  GAP();
  for (unsigned t = 0; t < 16u; ++t) {
    group<0>(s, make_iseq<8>{});
    s.ro = ro;
    if (__builtin_expect(known < t + 2u, 0)) {
      asm volatile("" : "+v"(s.fv));
      require(t + 2u);
    }
    GAP();
    group<1>(s, make_iseq<8>{});
  }
  STAMP(10);
  asm volatile("s_nop 15" ::: "memory");
  GAP();
  finish_all<1>(s, make_iseq<40>{});
  if (tid == 0) __hip_atomic_store(cnt, call, __ATOMIC_RELAXED, __HIP_MEMORY_SCOPE_AGENT);
  STAMP(11);
}

}

extern "C" void kernel_launch(int rank, int nranks, ncclComm_t comm,
                              hipStream_t stream,
                              void* const* d_in, const int* in_sizes, int n_in,
                              void* d_out, int out_size,
                              const int* in_dtypes, int out_dtype,
                              void* const* d_out_all,
                              void* const* d_ws_all, size_t ws_bytes) {
  (void)comm; (void)in_dtypes; (void)out_dtype; (void)d_out_all;
  if (nranks != NR || (unsigned)rank >= (unsigned)NR || n_in < 4 || ws_bytes < WS_NEED ||
      in_sizes[0] != MSH * KDIM || in_sizes[1] != KDIM * NLOC || out_size != NR * MSH * NLOC) {
    fprintf(stderr, "kernel_launch: unexpected problem shape\n");
    return;
  }
  Params p;
  for (int r = 0; r < NR; ++r) p.ws[r] = (char*)d_ws_all[r];
  p.x = (const unsigned char*)d_in[0];
  p.w = (const unsigned char*)d_in[1];
  p.sx = (const float*)d_in[2];
  p.sw = (const float*)d_in[3];
  p.out = (float*)d_out;
  p.myws = p.ws[rank];
  p.rank = rank;
  hipLaunchKernelGGL(ag_gemm_kernel, dim3(NWG), dim3(256), 0, stream, p);
  HIP_CHECK(hipGetLastError());
}
```

```cpp
#include <hip/hip_runtime.h>
#include <rccl/rccl.h>
#include <cstdio>
#include <utility>

#define HIP_CHECK(expr) do { hipError_t _e = (expr); if (_e != hipSuccess) { \
    fprintf(stderr, "HIP error %s at %s:%d\n", hipGetErrorString(_e), __FILE__, __LINE__); \
    return; } } while (0)

#define AG_TL 800

namespace {

constexpr int NR = 8;
constexpr int KDIM = 4096, NLOC = 2048, MSH = 512;
constexpr unsigned SHARD_BYTES = MSH * KDIM;
constexpr unsigned W_BYTES = KDIM * NLOC;
constexpr unsigned OUT_BYTES = NR * MSH * NLOC * 4u;
constexpr int NWG = 256;
constexpr int C_PUSH = 4, C_PF = 2, C_SVC = C_PUSH + C_PF;
constexpr int NPUSH = C_PUSH * 8;
constexpr int NCOPY = C_PF * 8;
constexpr int CW = 80;
constexpr int XD = 5;
constexpr int PW = 5;
constexpr int TD = 3;
constexpr unsigned TL = AG_TL, TL_OWN = 1250;
constexpr int WT = 17;
constexpr size_t WS_TMO = 0;
constexpr size_t WS_CALL = 4096;
constexpr size_t WS_FLAGS = 64 * 1024;
constexpr size_t WS_STAMPS = 256 * 1024;
constexpr size_t WS_COPY = 2u << 20;
constexpr size_t WS_STAGE = WS_COPY + 2 * (size_t)SHARD_BYTES;
constexpr size_t WS_NEED = WS_STAGE + 2 * NR * (size_t)SHARD_BYTES;
constexpr int NS = 48;
constexpr int NSTG = NS / 16;
static_assert(NS % 16 == 0 && NS >= 32 && NS <= 96, "push / copy split");
constexpr unsigned long long DEADLINE_TICKS = 50000000ull;

constexpr int KB_LDS = 128 * CW;
constexpr int WSLOTS = 3;
constexpr int WAVE_LDS = WSLOTS * KB_LDS;
constexpr int PRS = 272;
constexpr int P4OFF = 16 * PRS;
constexpr int PBUF = 5632;
constexpr int LDS_CTRL = 4 * WAVE_LDS;
constexpr int LDS_BYTES = LDS_CTRL + 64;

typedef float f32x4 __attribute__((ext_vector_type(4)));
typedef int i32x8 __attribute__((ext_vector_type(8)));
typedef int i32x2 __attribute__((ext_vector_type(2)));
typedef int i32x4 __attribute__((ext_vector_type(4)));
typedef unsigned u32x4 __attribute__((ext_vector_type(4)));
#define LAS __attribute__((address_space(3)))
#define GAS __attribute__((address_space(1)))
#define DI __device__ __forceinline__
#define GAP() __builtin_amdgcn_sched_barrier(0)
template <int... Is> using iseq = std::integer_sequence<int, Is...>;
template <int N> using make_iseq = std::make_integer_sequence<int, N>;

#define ACLOB "a0","a1","a2","a3","a4","a5","a6","a7","a8","a9","a10","a11","a12","a13","a14","a15","a16","a17","a18","a19","a20","a21","a22","a23","a24","a25","a26","a27","a28","a29","a30","a31","a32","a33","a34","a35","a36","a37","a38","a39","a40","a41","a42","a43","a44","a45","a46","a47","a48","a49","a50","a51","a52","a53","a54","a55","a56","a57","a58","a59","a60","a61","a62","a63","a64","a65","a66","a67","a68","a69","a70","a71","a72","a73","a74","a75","a76","a77","a78","a79","a80","a81","a82","a83","a84","a85","a86","a87","a88","a89","a90","a91","a92","a93","a94","a95","a96","a97","a98","a99","a100","a101","a102","a103","a104","a105","a106","a107","a108","a109","a110","a111","a112","a113","a114","a115","a116","a117","a118","a119","a120","a121","a122","a123","a124","a125","a126","a127", \
  "a128","a129","a130","a131","a132","a133","a134","a135","a136","a137","a138","a139","a140","a141","a142","a143","a144","a145","a146","a147","a148","a149","a150","a151","a152","a153","a154","a155","a156","a157","a158","a159","a160","a161","a162","a163","a164","a165","a166","a167","a168","a169","a170","a171","a172","a173","a174","a175","a176","a177","a178","a179","a180","a181","a182","a183","a184","a185","a186","a187","a188","a189","a190","a191","a192","a193","a194","a195","a196","a197","a198","a199","a200","a201","a202","a203","a204","a205","a206","a207","a208","a209","a210","a211","a212","a213","a214","a215","a216","a217","a218","a219","a220","a221","a222","a223","a224","a225","a226","a227","a228","a229","a230","a231","a232","a233","a234","a235","a236","a237","a238","a239","a240","a241","a242","a243","a244","a245","a246","a247","a248","a249","a250","a251","a252","a253","a254","a255"

struct Params {
  char* ws[NR];
  const unsigned char* x;
  const unsigned char* w;
  const float* sx;
  const float* sw;
  float* out;
  char* myws;
  int rank;
};

template <int N> DI void wait_vm() { asm volatile("s_waitcnt vmcnt(%0)" :: "n"(N) : "memory"); }
DI void wait_lgkm0() { asm volatile("s_waitcnt lgkmcnt(0)" ::: "memory"); }
DI unsigned long long rtc() { return __builtin_amdgcn_s_memrealtime(); }
DI unsigned rfl(unsigned v) { return __builtin_amdgcn_readfirstlane(v); }

struct St {
  __amdgpu_buffer_rsrc_t rx, rx2, rw, ro;
  unsigned vx;
  unsigned sx;
  unsigned vw[5];
  unsigned sw;
  int rot;
  unsigned vt, st;
  unsigned ldsw;
  unsigned tr;
  unsigned pw, pw4;
  unsigned pr, prb, pr4, pr4b;
  unsigned vo[2];
  unsigned so;
  i32x8 wv[16];
  f32x4 acc[2][5];
  f32x4 pt[4];
  float p4[4];
  unsigned scale_bits;
  __amdgpu_buffer_rsrc_t rf;
  unsigned vf, fv;
  unsigned sqw, sqr, sqv, sqn;
};

DI void w_dma(St& s, int kb, int slot) {
#pragma unroll
  for (int i = 0; i < 10; ++i) {
    const unsigned m0v = s.ldsw + slot * KB_LDS + i * 1024;
    const unsigned so = s.sw + (unsigned)(((kb + s.rot) & 7) * 128 + (i / 5) * 64) * NLOC;
    asm volatile("s_mov_b32 m0, %0\n\ts_nop 0\n\tbuffer_load_dwordx4 %1, %2, %3 offen lds" :: "s"(m0v), "v"(s.vw[i % 5]), "s"(s.rw), "s"(so) : "memory");
  }
}
DI void w_touch(St& s, int kb) {
  const unsigned so = s.st + (unsigned)(((kb + s.rot) & 7) * 128) * NLOC;
  asm volatile("buffer_load_dword a0, %0, %1, %2 offen" :: "v"(s.vt), "s"(s.rw), "s"(so) : "memory", "a0");
}
template <int F, int SLOT> DI void w_tr(St& s) {
  constexpr int O = SLOT * KB_LDS + (F % 5) * 16;
  if constexpr (F < 24) {
    constexpr int A = 64 + 8 * F;
    asm volatile("ds_read_b64_tr_b8 a[%c1:%c2], %0 offset:%c9\n\t"
                 "ds_read_b64_tr_b8 a[%c3:%c4], %0 offset:%c10\n\t"
                 "ds_read_b64_tr_b8 a[%c5:%c6], %0 offset:%c11\n\t"
                 "ds_read_b64_tr_b8 a[%c7:%c8], %0 offset:%c12"
                 :: "v"(s.tr), "i"(A), "i"(A + 1), "i"(A + 2), "i"(A + 3), "i"(A + 4), "i"(A + 5), "i"(A + 6), "i"(A + 7),
                    "i"(O), "i"(O + 8 * CW), "i"(O + 16 * CW), "i"(O + 24 * CW)
                 : "memory", ACLOB);
  } else {
    const unsigned base = s.tr + O;
    i32x2 r0 = __builtin_amdgcn_ds_read_tr8_b64_v2i32((LAS i32x2*)(base));
    i32x2 r1 = __builtin_amdgcn_ds_read_tr8_b64_v2i32((LAS i32x2*)(base + 8 * CW));
    i32x2 r2 = __builtin_amdgcn_ds_read_tr8_b64_v2i32((LAS i32x2*)(base + 16 * CW));
    i32x2 r3 = __builtin_amdgcn_ds_read_tr8_b64_v2i32((LAS i32x2*)(base + 24 * CW));
    i32x4 lo = __builtin_shufflevector(r0, r1, 0, 1, 2, 3), hi = __builtin_shufflevector(r2, r3, 0, 1, 2, 3);
    s.wv[F - 24] = __builtin_shufflevector(lo, hi, 0, 1, 2, 3, 4, 5, 6, 7);
  }
}
DI void f_sample(St& s) { asm volatile("buffer_load_dword %0, %1, %2, 0 offen sc0 sc1" : "=v"(s.fv) : "v"(s.vf), "s"(s.rf) : "memory"); }
template <int KB> DI void w_stage(St& s) {
  constexpr int LEFT = (7 - KB) < (WSLOTS - 1) ? (7 - KB) : (WSLOTS - 1);
  wait_vm<10 * LEFT>();
  if constexpr (KB + TD < 8) w_touch(s, KB + TD);
  w_tr<5 * KB + 0, KB % WSLOTS>(s); w_tr<5 * KB + 1, KB % WSLOTS>(s); w_tr<5 * KB + 2, KB % WSLOTS>(s);
  w_tr<5 * KB + 3, KB % WSLOTS>(s); w_tr<5 * KB + 4, KB % WSLOTS>(s);
  GAP();
  wait_lgkm0();
  if constexpr (KB + WSLOTS == 7) f_sample(s);
  if constexpr (KB + WSLOTS < 8) w_dma(s, KB + WSLOTS, KB % WSLOTS);
  GAP();
}
template <int... Is> DI void w_stage_all(St& s, iseq<Is...>) { (w_stage<Is>(s), ...); }

template <int SLOT> DI void x_load(St& s) {
  constexpr int A = 8 * SLOT;
  asm volatile("buffer_load_dwordx4 a[%c3:%c4], %0, %1, %2 offen\n\t"
               "buffer_load_dwordx4 a[%c5:%c6], %0, %1, %2 offen offset:1024"
               :: "v"(s.vx), "s"(s.rx), "s"(s.sx), "i"(A), "i"(A + 3), "i"(A + 4), "i"(A + 7) : "memory", ACLOB);
  s.sx += 8192u;
}
template <int... Is> DI void x_fill(St& s, iseq<Is...>) { (x_load<Is>(s), ...); }
template <int SET, int KB, int U> DI void mma(St& s) {
  constexpr int X = 8 * KB, F = 5 * KB + U;
  if constexpr (F < 24) {
    constexpr int A = 64 + 8 * F;
    if constexpr (KB == 0) asm volatile("v_mfma_f32_16x16x128_f8f6f4 %0, a[%c1:%c2], a[%c3:%c4], 0" : "=v"(s.acc[SET][U]) : "i"(A), "i"(A + 7), "i"(X), "i"(X + 7));
    else asm volatile("v_mfma_f32_16x16x128_f8f6f4 %0, a[%c1:%c2], a[%c3:%c4], %0" : "+v"(s.acc[SET][U]) : "i"(A), "i"(A + 7), "i"(X), "i"(X + 7));
  } else {
    asm volatile("v_mfma_f32_16x16x128_f8f6f4 %0, %1, a[%c2:%c3], %0" : "+v"(s.acc[SET][U]) : "v"(s.wv[F - 24]), "i"(X), "i"(X + 7));
  }
}
DI void red_barrier() { wait_lgkm0(); __builtin_amdgcn_s_barrier(); asm volatile("" ::: "memory"); }
DI float fadd(float a, float b) { float r; asm("v_add_f32_e32 %0, %1, %2" : "=v"(r) : "v"(a), "v"(b)); return r; }
DI float fscale(unsigned sb, float a) { float r; asm("v_mul_f32_e32 %0, %1, %2" : "=v"(r) : "s"(sb), "v"(a)); return r; }
DI float frelu(float a) { return __builtin_elementwise_maximum(a, 0.f); }
template <int P, int V> DI void red_op(St& s) {
  constexpr int e = V & 3;
  if constexpr (V < 4) s.pt[0][e] = fadd(s.pt[0][e], s.pt[1][e]);
  else if constexpr (V < 8) s.pt[2][e] = fadd(s.pt[2][e], s.pt[3][e]);
  else if constexpr (V < 12) s.pt[0][e] = fadd(s.pt[0][e], s.pt[2][e]);
  else if constexpr (V < 16) s.pt[0][e] = fscale(s.scale_bits, s.pt[0][e]);
  else if constexpr (V < 20) s.pt[0][e] = frelu(s.pt[0][e]);
  else if constexpr (V == 20) s.p4[0] = fadd(s.p4[0], s.p4[1]);
  else if constexpr (V == 21) s.p4[2] = fadd(s.p4[2], s.p4[3]);
  else if constexpr (V == 22) s.p4[0] = fadd(s.p4[0], s.p4[2]);
  else if constexpr (V == 23) s.p4[0] = fscale(s.scale_bits, s.p4[0]);
  else if constexpr (V == 24) s.p4[0] = frelu(s.p4[0]);
}
struct Sched { int w[5], chk, r[4], r4[2], v0; };
constexpr Sched SC = {{1, 2, 3, 4, 6}, 12, {13, 14, 16, 17}, {18, 19}, 21};
constexpr int vslot(int G) { int n = 0; for (int g = SC.v0; g < G; ++g) n += (g % 5 != 0); return n; }
template <int P, int G, bool TAIL = false> DI void finish(St& s) {
  if constexpr (G == SC.w[0]) *(LAS f32x4*)(s.pw + P * PBUF) = s.acc[P][0];
  if constexpr (G == SC.w[1]) *(LAS f32x4*)(s.pw + P * PBUF + 64) = s.acc[P][1];
  if constexpr (G == SC.w[2]) *(LAS f32x4*)(s.pw + P * PBUF + 128) = s.acc[P][2];
  if constexpr (G == SC.w[3]) *(LAS f32x4*)(s.pw + P * PBUF + 192) = s.acc[P][3];
  if constexpr (G == SC.w[4]) *(LAS f32x4*)(s.pw4 + P * PBUF) = s.acc[P][4];
  if constexpr (TAIL && G == SC.chk) red_barrier();
  if constexpr (!TAIL && G == SC.w[4] + 1) { asm("v_add_u32_e32 %0, 1, %1" : "=v"(s.sqn) : "v"(s.sqn)); *(volatile LAS unsigned*)(s.sqw) = s.sqn; }
  if constexpr (!TAIL && G == SC.chk) { s.sqv = *(volatile LAS unsigned*)(s.sqr); asm volatile("" ::: "memory"); }
  if constexpr (!TAIL && G == SC.v0) {
    if (__builtin_expect(__builtin_amdgcn_ballot_w64(s.sqv < s.sqn) != 0, 0)) {
      do { s.sqv = *(volatile LAS unsigned*)(s.sqr); } while (__builtin_amdgcn_ballot_w64(s.sqv < s.sqn) != 0);
      asm volatile("" ::: "memory");
      s.pt[0] = *(LAS f32x4*)(s.pr + P * PBUF); s.pt[1] = *(LAS f32x4*)(s.pr + P * PBUF + WAVE_LDS);
      s.pt[2] = *(LAS f32x4*)(s.prb + P * PBUF); s.pt[3] = *(LAS f32x4*)(s.prb + P * PBUF + WAVE_LDS);
      s.p4[0] = *(LAS float*)(s.pr4 + P * PBUF); s.p4[1] = *(LAS float*)(s.pr4 + P * PBUF + WAVE_LDS);
      s.p4[2] = *(LAS float*)(s.pr4b + P * PBUF); s.p4[3] = *(LAS float*)(s.pr4b + P * PBUF + WAVE_LDS);
    }
  }
  if constexpr (G == SC.r[0]) s.pt[0] = *(LAS f32x4*)(s.pr + P * PBUF);
  if constexpr (G == SC.r[1]) s.pt[1] = *(LAS f32x4*)(s.pr + P * PBUF + WAVE_LDS);
  if constexpr (G == SC.r[2]) s.pt[2] = *(LAS f32x4*)(s.prb + P * PBUF);
  if constexpr (G == SC.r[3]) s.pt[3] = *(LAS f32x4*)(s.prb + P * PBUF + WAVE_LDS);
  if constexpr (G == SC.r4[0]) { s.p4[0] = *(LAS float*)(s.pr4 + P * PBUF); s.p4[1] = *(LAS float*)(s.pr4 + P * PBUF + WAVE_LDS); }
  if constexpr (G == SC.r4[1]) { s.p4[2] = *(LAS float*)(s.pr4b + P * PBUF); s.p4[3] = *(LAS float*)(s.pr4b + P * PBUF + WAVE_LDS); }
  if constexpr (G >= SC.v0 && G % 5 != 0) {
    constexpr int n = vslot(G);
    if constexpr (n < 10) { red_op<P, 2 * n>(s); red_op<P, 2 * n + 1>(s); }
    if constexpr (n == 10) { __builtin_amdgcn_raw_buffer_store_b128(__builtin_bit_cast(u32x4, s.pt[0]), s.ro, s.vo[0], s.so, WT); red_op<P, 20>(s); }
    if constexpr (n == 11) { red_op<P, 21>(s); red_op<P, 22>(s); asm volatile("" :: "v"(s.pt[0])); }
    if constexpr (n == 12) { red_op<P, 23>(s); red_op<P, 24>(s); }
    if constexpr (n == 13) { __builtin_amdgcn_raw_buffer_store_b32(__float_as_uint(s.p4[0]), s.ro, s.vo[1], s.so, WT); s.so += 16u * NLOC * 4u; }
  }
}
static_assert(vslot(40) > 13, "the finish must fit one group");
template <int P, int... Gs> DI void finish_all(St& s, iseq<Gs...>) { (finish<P, Gs, true>(s), ...); }
constexpr int SAMPLE_G = 7;
template <int SET, int KB, int U> DI void gap(St& s) {
  mma<SET, KB, U>(s);
  if constexpr (U == 0) {
    if constexpr (SET == 1 && KB == 8 - XD) { if (s.sx == NS * 8192u) s.rx = s.rx2; }
    x_load<(KB + XD) & 7>(s);
  }
  if constexpr (SET == 0 && 5 * KB + U == SAMPLE_G) f_sample(s);
  finish<SET ^ 1, 5 * KB + U>(s);
  GAP();
}
static_assert(XD >= 2 && 5 * (7 - XD) > SAMPLE_G, "ring of 8 slots; the flag sample must retire inside its group");
constexpr int gap_of(int n) { for (int g = SC.v0; g < 40; ++g) if (g % 5 != 0 && vslot(g) == n) return g; return -1; }
constexpr int younger_other(int set, int kb) {
  int c = 0;
  for (int p = 5 * (kb - XD); p < 5 * kb; ++p) {
    const int g = p < 0 ? p + 40 : p, st = p < 0 ? set ^ 1 : set;
    c += (g == gap_of(10)) + (g == gap_of(13)) + (st == 0 && g == SAMPLE_G);
  }
  return c;
}
template <int SET, int KB> DI void step(St& s) {
  wait_vm<2 * (XD - 1) + younger_other(SET, KB)>();
  gap<SET, KB, 0>(s); gap<SET, KB, 1>(s); gap<SET, KB, 2>(s); gap<SET, KB, 3>(s); gap<SET, KB, 4>(s);
}
template <int SET, int... Is> DI void group(St& s, iseq<Is...>) { (step<SET, Is>(s), ...); }

DI unsigned flags_known(unsigned f, unsigned call) {
  unsigned k = 0;
  while (k < NSTG && (unsigned)__ballot(f >= call * 8u + k + 1u) == 0xffffffffu) ++k;
  const bool cp = ((__ballot(f >= call * 8u + 7u) >> NPUSH) & ((1ull << NCOPY) - 1)) == (1ull << NCOPY) - 1;
  return (cp && k == NSTG) ? 99u : k;
}
DI unsigned wait_flags(char* myws, int src, unsigned stage, int npush, bool copy, unsigned call, int lane) {
  GAS unsigned* fl = (GAS unsigned*)(myws + WS_FLAGS) + src * 64 + lane;
  GAS unsigned* tmo = (GAS unsigned*)(myws + WS_TMO);
  const unsigned long long t0 = rtc();
  for (unsigned it = 0;; ++it) {
    const unsigned f = __hip_atomic_load(fl, __ATOMIC_RELAXED, __HIP_MEMORY_SCOPE_SYSTEM);
    const unsigned long long m = __ballot(f >= call * 8u + stage);
    const bool cp = ((m >> NPUSH) & ((1ull << NCOPY) - 1)) == (1ull << NCOPY) - 1;
    if ((!copy || cp) && __builtin_popcount((unsigned)m) >= npush) { const unsigned k = flags_known(f, call); return k ? k : 1u; }
    if ((it & 63u) == 63u) {
      if (rfl(__hip_atomic_load(tmo, __ATOMIC_RELAXED, __HIP_MEMORY_SCOPE_AGENT)) == call || rtc() - t0 > DEADLINE_TICKS) {
        if (lane == 0) __hip_atomic_store(tmo, call, __ATOMIC_RELAXED, __HIP_MEMORY_SCOPE_AGENT);
        return 0u;
      }
    }
    __builtin_amdgcn_s_sleep(1);
  }
}
DI void unit_map(int cb, unsigned& base, unsigned& pack) {
  const unsigned l0 = (unsigned)(cb - 16) * 5u, q = (l0 * 43u) >> 7, r = l0 - 3u * q;
  base = cb < 16 ? (unsigned)cb * 8u : q * 8u + 5u + r;
  pack = cb < 16 ? 0x43210u : r == 0 ? 0x98210u : r == 1 ? 0x98710u : 0xe8760u;
}
static_assert(NPUSH == 32 && NPUSH + NCOPY <= 64, "flag word layout");
DI GAS unsigned* flag_dst(char* const (&pb)[NR], int word) {
  unsigned lo = 0, hi = 0;
#pragma unroll
  for (int i = 0; i < NR; ++i) {
    const size_t a = (size_t)pb[i] + WS_FLAGS + (size_t)word * 4;
    asm("v_writelane_b32 %0, %2, %c4\n\tv_writelane_b32 %1, %3, %c4" : "+v"(lo), "+v"(hi) : "s"((unsigned)a), "s"((unsigned)(a >> 32)), "i"(i));
  }
  return (GAS unsigned*)(((size_t)hi << 32) | lo);
}
DI unsigned unit_src(int u, int rank, int lane) {
  const int pos = u >> 3, q = (u >> 1) & 3, h = u & 1;
  return (unsigned)(((pos >> 3) * 16 + (lane & 15)) * KDIM + q * 1024 + ((pos + rank) & 7) * 128 + (lane >> 4) * 32 + h * 16);
}

#define STAMP(i) do { } while (0)

__global__ void __launch_bounds__(256, 1) ag_gemm_kernel(Params p) {
  __shared__ __attribute__((aligned(1024))) unsigned char lds[LDS_BYTES];
  const int tid = threadIdx.x, lane = tid & 63;
  const int wave = __builtin_amdgcn_readfirstlane(tid >> 6);
  const int b = blockIdx.x, src = b & 7, c = b >> 3;
  int rank = p.rank & 7;
  char* myws = p.myws;
  const unsigned char* xp = p.x;
  const unsigned char* wp = p.w;
  const float* sxp = p.sx;
  const float* swp = p.sw;
  char* pb[NR];
#pragma unroll
  for (int i = 0; i < NR; ++i) pb[i] = p.ws[(b * 4 + wave + i) & 7];
  asm volatile("" : "+s"(rank), "+s"(myws), "+s"(xp), "+s"(wp), "+s"(sxp), "+s"(swp), "+s"(pb[0]), "+s"(pb[1]), "+s"(pb[2]), "+s"(pb[3]), "+s"(pb[4]), "+s"(pb[5]), "+s"(pb[6]), "+s"(pb[7]));
  GAS unsigned* const cnt = (GAS unsigned*)(myws + WS_CALL) + b * 16;
  STAMP(0);

  if (__builtin_expect(c < C_PUSH, 1)) {
    const int pw = c * 8 + src, w = pw * 4 + wave;
    u32x4 v[NSTG];
    const GAS unsigned char* xs = (const GAS unsigned char*)xp + unit_src(w, rank, lane);
    v[0] = *(const GAS u32x4*)xs;
    const unsigned cv = __hip_atomic_load(cnt, __ATOMIC_RELAXED, __HIP_MEMORY_SCOPE_AGENT);
#pragma unroll
    for (int i = 1; i < NSTG; ++i) v[i] = *(const GAS u32x4*)(xs + (size_t)i * 32 * KDIM);
    __amdgpu_buffer_rsrc_t rd[NR];
#pragma unroll
    for (int d = 0; d < NR; ++d) rd[d] = __builtin_amdgcn_make_buffer_rsrc(pb[d] + WS_STAGE + (size_t)rank * SHARD_BYTES, 0, (int)(NR * SHARD_BYTES) + NS * 8192, 0x00020000);
    GAS unsigned* fdst = flag_dst(pb, rank * 64 + pw);
    GAP();
    const unsigned call = rfl(cv) + 1u;
    const unsigned so = (call & 1u) * (NR * SHARD_BYTES) + (unsigned)w * 1024u;
    static_assert(PW <= NR, "only the last stage is published behind the last store");
#pragma unroll
    for (int n = 0; n <= NSTG * NR; ++n) {
      const int done = n < NSTG * NR ? n - PW + 1 : n;
      if (n == NSTG * NR) wait_vm<0>(); else if (n >= PW) wait_vm<PW - 1>();
      if (done > 0 && done % NR == 0) {
        __builtin_amdgcn_s_barrier();
        if (tid < NR) __hip_atomic_store(fdst, call * 8u + (unsigned)(done / NR), __ATOMIC_RELAXED, __HIP_MEMORY_SCOPE_SYSTEM);
        if (done / NR == 2) STAMP(2);
      }
      if (n < NSTG * NR) __builtin_amdgcn_raw_buffer_store_b128(v[n / NR], rd[n % NR], (unsigned)(lane * 16), so + (unsigned)(n / NR) * 131072u, WT);
      if (n == 0) STAMP(1);
    }
    if (tid == 0) __hip_atomic_store(cnt, call, __ATOMIC_RELAXED, __HIP_MEMORY_SCOPE_AGENT);
    STAMP(3);
    return;
  }
  char* sws = p.ws[src];
  if (c < C_SVC) {
    const unsigned long long tstart = rtc();
    if (wave == 0 && c == C_PUSH) __builtin_amdgcn_fence(__ATOMIC_ACQUIRE, "");
    const int cw = (c - C_PUSH) * 8 + src, u0 = NS * 8 + (cw * 4 + wave) * 2;
    constexpr int U = 16 - NS / 16;
    u32x4 v[2 * U];
    const GAS unsigned char* xs = (const GAS unsigned char*)xp + unit_src(u0, rank, lane);
    const unsigned cv = __hip_atomic_load(cnt, __ATOMIC_RELAXED, __HIP_MEMORY_SCOPE_AGENT);
#pragma unroll
    for (int i = 0; i < U; ++i) { v[2 * i] = *(const GAS u32x4*)(xs + (size_t)i * 32 * KDIM); v[2 * i + 1] = *(const GAS u32x4*)(xs + (size_t)i * 32 * KDIM + 16); }
    const unsigned call = rfl(cv) + 1u;
    __amdgpu_buffer_rsrc_t rc = __builtin_amdgcn_make_buffer_rsrc(myws + WS_COPY + (size_t)(call & 1u) * SHARD_BYTES, 0, (int)SHARD_BYTES, 0x00020000);
#pragma unroll
    for (int i = 0; i < 2 * U; ++i) __builtin_amdgcn_raw_buffer_store_b128(v[i], rc, (unsigned)(lane * 16 + (i & 1) * 1024), (unsigned)((u0 + 128 * (i >> 1)) * 1024), WT);
    GAS unsigned* fdst = flag_dst(pb, rank * 64 + NPUSH + cw);
    wait_vm<0>();
    __builtin_amdgcn_s_barrier();
    if (tid < NR) __hip_atomic_store(fdst, call * 8u + 7u, __ATOMIC_RELAXED, __HIP_MEMORY_SCOPE_SYSTEM);
    if (tid == 0) __hip_atomic_store(cnt, call, __ATOMIC_RELAXED, __HIP_MEMORY_SCOPE_AGENT);
    STAMP(1);
    __amdgpu_buffer_rsrc_t rs = __builtin_amdgcn_make_buffer_rsrc(sws + WS_COPY + (size_t)(call & 1u) * SHARD_BYTES, 0, (int)SHARD_BYTES, 0x00020000);
    const unsigned vo = (unsigned)((c - C_PUSH) * 4096 + wave * 1024 + lane * 16);
    if (wave == 0) *(LAS unsigned*)lds = wait_flags(myws, src, 1, 0, true, call, lane);
    red_barrier();
    if (!rfl(*(LAS unsigned*)lds)) return;
    while (rtc() - tstart < (src == rank ? TL_OWN : TL)) __builtin_amdgcn_s_sleep(1);
    STAMP(2);
    static_assert(C_PF == 2, "two 4 KiB halves per position");
    for (unsigned so = NS * 8192u; so < SHARD_BYTES; so += 8192u) {
      wait_vm<6>();
      asm volatile("buffer_load_dwordx4 a[0:3], %0, %1, %2 offen" :: "v"(vo), "s"(rs), "s"(so) : "memory", "a0", "a1", "a2", "a3");
      if (so & 8192u) __builtin_amdgcn_s_barrier();
    }
    wait_vm<0>();
    STAMP(3);
    return;
  }

  const int cb = c - C_SVC;
  const unsigned ldsbase = (unsigned)(size_t)(LAS unsigned char*)lds;
  St s;
  unsigned cv, sxv, swv;
  {
    const unsigned zero = 0;
    asm volatile("global_load_dword %0, %3, %4 sc1\n\tglobal_load_dword %1, %3, %5\n\tglobal_load_dword %2, %3, %6"
                 : "=&v"(cv), "=&v"(sxv), "=&v"(swv) : "v"(zero), "s"(cnt), "s"(sxp), "s"(swp) : "memory");
  }
  s.ldsw = ldsbase + wave * WAVE_LDS;
  s.rw = __builtin_amdgcn_make_buffer_rsrc((void*)wp, 0, (int)W_BYTES, 0x00020000);
  unsigned ubase, upack;
  unit_map(cb, ubase, upack);
#pragma unroll
  for (int i = 0; i < 5; ++i) {
    const unsigned e = (unsigned)(i * 64 + lane), q = (e * 205u) >> 10, r = e - 5u * q;
    const unsigned un = ubase + ((upack >> (4u * r)) & 15u);
    s.vw[i] = q * NLOC + un * 16u + (un >> 7) * 0x40000000u;
  }
  s.sw = (unsigned)(wave * 1024 * NLOC);
  s.vt = (unsigned)(lane * NLOC);
  {
    const unsigned l3 = (3u - (unsigned)((size_t)wp >> 7)) & 7u;
    s.st = ((unsigned)((wave * 1024 + ((cb >> 2) & 1) * 64) * NLOC) + (l3 + ((cb >> 3) & 1) * 8u) * 128u) | ((cb < 16 && wave == (cb & 3)) ? 0u : 0x40000000u);
  }
  s.rot = src;
  for (int kb = 1; kb < TD; ++kb) w_touch(s, kb);
  STAMP(3);
  for (int kb = 0; kb < WSLOTS; ++kb) w_dma(s, kb, kb);
  GAP();
  s.tr = s.ldsw + (unsigned)((32 * (lane >> 4) + ((lane & 15) >> 1)) * CW + (lane & 1) * 8);
  s.vx = (unsigned)(wave * 2048 + lane * 16);
  s.sx = 0;
  s.pw = s.ldsw + (unsigned)((lane & 15) * PRS + (lane >> 4) * 16);
  s.pw4 = s.ldsw + (unsigned)(P4OFF + lane * 16 + (lane >> 4) * 32);
  {
    const int row = wave * 4 + (lane >> 4), col = lane & 15;
    s.pr = ldsbase + (unsigned)(row * PRS + col * 16);
    s.prb = s.pr + 2 * WAVE_LDS;
    s.pr4 = ldsbase + (unsigned)(P4OFF + row * 16 + (col >> 2) * (256 + 32) + (col & 3) * 4);
    s.pr4b = s.pr4 + 2 * WAVE_LDS;
    asm volatile("" : "+v"(s.pr), "+v"(s.prb), "+v"(s.pr4), "+v"(s.pr4b), "+v"(s.pw), "+v"(s.pw4));
    const unsigned u0 = ubase + ((upack >> (4 * (col >> 2))) & 15u), u4 = ubase + (upack >> 16);
    s.vo[0] = u0 < 128 ? (unsigned)(row * NLOC * 4 + u0 * 64 + (col & 3) * 16) : 0x7ff00000u;
    s.vo[1] = u4 < 128 ? (unsigned)(row * NLOC * 4 + u4 * 64 + col * 4) : 0x7ff00000u;
  }
  const __amdgpu_buffer_rsrc_t ro = __builtin_amdgcn_make_buffer_rsrc((void*)p.out, 0, (int)OUT_BYTES, 0x00020000);
  s.ro = __builtin_amdgcn_make_buffer_rsrc((void*)p.out, 0, 0, 0x00020000);
  s.so = (unsigned)(src * MSH - 16) * (unsigned)(NLOC * 4);
  s.rf = __builtin_amdgcn_make_buffer_rsrc(myws + WS_FLAGS + src * 256, 0, wave == 0 ? 256 : 0, 0x00020000);
  s.vf = (unsigned)(lane * 4);
  s.sqw = ldsbase + LDS_CTRL + 16 + wave * 4;
  s.sqr = ldsbase + LDS_CTRL + 16 + (lane & 3) * 4;
  s.sqn = 0;
  asm volatile("" : "+v"(s.sqw), "+v"(s.sqr), "+v"(s.sqn));
  *(volatile LAS unsigned*)(s.sqw) = 0u;
#pragma unroll
  for (int u = 0; u < 5; ++u) s.acc[1][u] = f32x4{0.f, 0.f, 0.f, 0.f};
  w_stage_all(s, make_iseq<7>{});
  asm volatile("" : "+v"(cv), "+v"(sxv), "+v"(swv), "+v"(s.fv));
  const unsigned call = rfl(cv) + 1u;
  s.scale_bits = rfl(__float_as_uint(__uint_as_float(sxv) * __uint_as_float(swv)));
  s.rx2 = __builtin_amdgcn_make_buffer_rsrc(sws + WS_COPY + (size_t)(call & 1u) * SHARD_BYTES, 0, (int)SHARD_BYTES, 0x00020000);
  s.rx = __builtin_amdgcn_make_buffer_rsrc(myws + WS_STAGE + (size_t)((call & 1u) * NR + src) * SHARD_BYTES, 0, (int)SHARD_BYTES, 0x00020000);
  STAMP(1);
  unsigned known = 0;
  auto require = [&](unsigned need) {
    if (wave == 0) {
      unsigned k = flags_known(s.fv, call);
      if (k < need) {
        const bool last = need > NSTG;
        STAMP(4 + need);
        k = wait_flags(myws, src, last ? NSTG : need, NPUSH, last, call, lane);
      }
      *(LAS unsigned*)(ldsbase + LDS_CTRL) = k;
    }
    red_barrier();
    known = rfl(*(LAS unsigned*)(ldsbase + LDS_CTRL));
    if (!known) {
      s.rx = s.rx2 = __builtin_amdgcn_make_buffer_rsrc(myws, 0, 0, 0x00020000);
      s.scale_bits = 0x7fc00000u;
      known = 99u;
    }
    if (known == 99u) s.rf = __builtin_amdgcn_make_buffer_rsrc(myws, 0, 0, 0x00020000);
    __builtin_amdgcn_s_waitcnt(0xC07F);
    asm volatile("s_nop 4" ::: "memory");
  };
  require(1u);
  x_fill(s, make_iseq<XD>{});
  GAP();
  wait_vm<2 * XD>();
  STAMP(2);
  w_tr<35, 7 % WSLOTS>(s); w_tr<36, 7 % WSLOTS>(s); w_tr<37, 7 % WSLOTS>(s); w_tr<38, 7 % WSLOTS>(s); w_tr<39, 7 % WSLOTS>(s);
  wait_vm<0>();
  __builtin_amdgcn_s_waitcnt(0xC07F);
  asm volatile("s_nop 1" ::: "memory");
  GAP();
  for (unsigned t = 0; t < 16u; ++t) {
    group<0>(s, make_iseq<8>{});
    s.ro = ro;
    if (__builtin_expect(known < t + 2u, 0)) {
      asm volatile("" : "+v"(s.fv));
      require(t + 2u);
    }
    GAP();
    group<1>(s, make_iseq<8>{});
  }
  STAMP(10);
  asm volatile("s_nop 15" ::: "memory");
  GAP();
  finish_all<1>(s, make_iseq<40>{});
  if (tid == 0) __hip_atomic_store(cnt, call, __ATOMIC_RELAXED, __HIP_MEMORY_SCOPE_AGENT);
  STAMP(11);
}

}

extern "C" void kernel_launch(int rank, int nranks, ncclComm_t comm,
                              hipStream_t stream,
                              void* const* d_in, const int* in_sizes, int n_in,
                              void* d_out, int out_size,
                              const int* in_dtypes, int out_dtype,
                              void* const* d_out_all,
                              void* const* d_ws_all, size_t ws_bytes) {
  (void)comm; (void)in_dtypes; (void)out_dtype; (void)d_out_all;
  if (nranks != NR || (unsigned)rank >= (unsigned)NR || n_in < 4 || ws_bytes < WS_NEED ||
      in_sizes[0] != MSH * KDIM || in_sizes[1] != KDIM * NLOC || out_size != NR * MSH * NLOC) {
    fprintf(stderr, "kernel_launch: unexpected problem shape\n");
    return;
  }
  Params p;
  for (int r = 0; r < NR; ++r) p.ws[r] = (char*)d_ws_all[r];
  p.x = (const unsigned char*)d_in[0];
  p.w = (const unsigned char*)d_in[1];
  p.sx = (const float*)d_in[2];
  p.sw = (const float*)d_in[3];
  p.out = (float*)d_out;
  p.myws = p.ws[rank];
  p.rank = rank;
  hipLaunchKernelGGL(ag_gemm_kernel, dim3(NWG), dim3(256), 0, stream, p);
  HIP_CHECK(hipGetLastError());
}
```
